```python
import numpy as np

kernel_name = "gpu_gemm_a2a_m4096_k4096_n16384_f32_none"


ABSMAX_RTOL = 0.001

COMPUTE_DTYPE = "float32"
EPILOGUE = "none"
IN_QUANT = None
OUT_QUANT = None
IN_DTYPES = {"x": "float32", "w_mat": "float32"}
WORLD_SIZE = 8

def setup_inputs():

    x = np.random.randn(4096, 4096).astype(np.float32)
    w_mat = (np.random.randn(4096, 16384) / np.sqrt(4096)).astype(np.float32)
    return {"x": x, "w_mat": w_mat}

def _epilogue(y):
    return y

def reference(x, w_mat):
    y = (x @ w_mat).astype(np.float32)
    return _epilogue(y)
```

```cpp
#include <hip/hip_runtime.h>
#include <rccl/rccl.h>
#include <cstdio>

#define HIP_CHECK(expr) do { hipError_t _e = (expr); if (_e != hipSuccess) { \
    fprintf(stderr, "HIP error %s at %s:%d\n", hipGetErrorString(_e), __FILE__, __LINE__); \
    return; } } while (0)

namespace {

constexpr int NR = 8;
constexpr int KDIM = 4096, N_ALL = 16384, N_LOC = 2048, M_LOC = 512;
constexpr int NST = 64;
constexpr unsigned A_TILE = 16384;
constexpr unsigned B_TILE = 32768;
constexpr unsigned SLOT = A_TILE + B_TILE;
constexpr int NSLOT = 3;
constexpr unsigned MiB = 1u << 20;

constexpr unsigned OFF_CNT = 0;
constexpr unsigned OFF_TMO = 32 * 1024;
constexpr unsigned OFF_XFLAG = 64 * 1024;
constexpr unsigned OFF_AFLAG = 128 * 1024;
constexpr unsigned OFF_WFLAG = 256 * 1024;
constexpr unsigned OFF_DBG = 512 * 1024;
constexpr unsigned OFF_XS = 1 * MiB;
constexpr unsigned OFF_XA = 9 * MiB;
constexpr unsigned OFF_WB = 41 * MiB;
constexpr size_t WS_NEED = 57ull * MiB;

constexpr unsigned L_FULL = NSLOT * SLOT;
constexpr unsigned L_FREE = L_FULL + 16;
constexpr unsigned L_XCNT = L_FREE + 16;
constexpr unsigned L_ABORT = L_XCNT + 4;
constexpr unsigned LDS_BYTES = L_FULL + 64;

constexpr unsigned long long DEADLINE_TICKS = 50000000ull;

typedef unsigned u32x4 __attribute__((ext_vector_type(4)));
typedef float f32x4 __attribute__((ext_vector_type(4)));
typedef float f32x2 __attribute__((ext_vector_type(2)));
typedef _Float16 f16x2 __attribute__((ext_vector_type(2)));
typedef _Float16 f16x8 __attribute__((ext_vector_type(8)));
#define LAS __attribute__((address_space(3)))

struct Params {
    const float* x;
    const float* w;
    float* out;
    char* ws[NR];
    int rank;
    int pad;
};

__device__ __forceinline__ unsigned pk(float a, float b) {
    f32x2 v = {a, b};
    return __builtin_bit_cast(unsigned, __builtin_convertvector(v, f16x2));
}
template <int N> __device__ __forceinline__ void waitvm() { asm volatile("s_waitcnt vmcnt(%0)" :: "i"(N) : "memory"); }
__device__ __forceinline__ const char* uni(const char* p) {
    unsigned long long v = (unsigned long long)p;
    unsigned lo = __builtin_amdgcn_readfirstlane((unsigned)v), hi = __builtin_amdgcn_readfirstlane((unsigned)(v >> 32));
    return (const char*)(((unsigned long long)hi << 32) | lo);
}
template <int IMM> __device__ __forceinline__ void ld16(f32x4& d, const char* base, unsigned voff) {
    asm volatile("global_load_dwordx4 %0, %1, %2 offset:%3" : "=v"(d) : "v"(voff), "s"(base), "i"(IMM) : "memory");
}
template <int IMM> __device__ __forceinline__ void ld16_sys(u32x4& d, const char* base, unsigned voff) {
    asm volatile("global_load_dwordx4 %0, %1, %2 offset:%3 sc0 sc1" : "+v"(d) : "v"(voff), "s"(base), "i"(IMM) : "memory");
}
template <int IMM> __device__ __forceinline__ void st16_sys(const char* base, unsigned voff, u32x4 v) {
    asm volatile("global_store_dwordx4 %0, %1, %2 offset:%3 sc0 sc1\n\ts_nop 1" :: "v"(voff), "v"(v), "s"(base), "i"(IMM) : "memory");
}
__device__ __forceinline__ void st4_sys(const char* base, unsigned voff, unsigned v) {
    asm volatile("global_store_dword %0, %1, %2 sc0 sc1" :: "v"(voff), "v"(v), "s"(base) : "memory");
}
__device__ __forceinline__ unsigned ld4_sys_wait(const char* base, unsigned voff) {
    unsigned v;
    asm volatile("global_load_dword %0, %1, %2 sc0 sc1\n\ts_waitcnt vmcnt(0)" : "=&v"(v) : "v"(voff), "s"(base) : "memory");
    return v;
}
__device__ __forceinline__ void glds4k(const char* base, unsigned voff, unsigned lds) {
    unsigned keep;
    asm volatile("s_mov_b32 %0, m0\n\ts_mov_b32 m0, %3\n\ts_nop 0\n\t"
                 "global_load_lds_dwordx4 %1, %2 sc0 sc1\n\t"
                 "global_load_lds_dwordx4 %1, %2 offset:1024 sc0 sc1\n\t"
                 "global_load_lds_dwordx4 %1, %2 offset:2048 sc0 sc1\n\t"
                 "global_load_lds_dwordx4 %1, %2 offset:3072 sc0 sc1\n\t"
                 "s_mov_b32 m0, %0"
                 : "=&s"(keep) : "v"(voff), "s"(base), "s"(lds) : "memory");
}
__device__ __forceinline__ void lds_st(unsigned addr, unsigned v) {
    asm volatile("ds_write_b32 %0, %1" :: "v"(addr), "v"(v) : "memory");
}
__device__ __forceinline__ unsigned lds_ld_wait(unsigned addr) {
    unsigned v;
    asm volatile("ds_read_b32 %0, %1\n\ts_waitcnt lgkmcnt(0)" : "=&v"(v) : "v"(addr) : "memory");
    return v;
}
__device__ __forceinline__ bool ge(unsigned v, unsigned want) { return (int)(v - want) >= 0; }

__device__ __forceinline__ void poll_all(const char* myws, unsigned voff, unsigned call) {
    if (__all(ge(ld4_sys_wait(myws, voff), call))) return;
    const unsigned long long t0 = __builtin_amdgcn_s_memrealtime();
    for (;;) {
        if (__all(ge(ld4_sys_wait(myws, voff), call))) return;
        if (__builtin_amdgcn_s_memrealtime() - t0 > 2000) {
            if ((unsigned)__builtin_amdgcn_readfirstlane(ld4_sys_wait(myws, OFF_TMO)) == call) return;
            if (__builtin_amdgcn_s_memrealtime() - t0 > DEADLINE_TICKS) {
                st4_sys(myws, OFF_TMO, call);
                waitvm<0>();
                return;
            }
        }
        __builtin_amdgcn_s_sleep(1);
    }
}

#define STAMP(i) do {} while (0)

__global__ __launch_bounds__(512, 2) void gemm_a2a_kernel(Params P) {
    extern __shared__ __attribute__((aligned(16))) char smem[];
    const int lane = threadIdx.x & 63;
    const int wave = __builtin_amdgcn_readfirstlane(threadIdx.x >> 6);
    const int b = blockIdx.x;
    const int rank = P.rank;
    const char* myws = uni(P.ws[rank]);
    const unsigned lds0 = (unsigned)(unsigned long long)(LAS char*)smem;
    const int q = lane >> 4, c = lane & 15;

    const int grp = b & 7, idx = b >> 3;
    const int src = idx >> 2;
    const int rb = grp >> 1;
    const int cb = (grp & 1) * 4 + (idx & 3);

    if (wave == 0) STAMP(0);
    f32x4 xr[8];
    if (wave >= 4) {
        const unsigned o = (unsigned)((((b & 3) * 128 + (wave - 4) * 32 + c) * KDIM + (b >> 2) * 64 + q * 4) * 4);
        const char* xb = uni((const char*)P.x);
        ld16<0>(xr[0], xb, o); ld16<64>(xr[1], xb, o); ld16<128>(xr[2], xb, o); ld16<192>(xr[3], xb, o);
        const unsigned o2 = o + 16 * KDIM * 4;
        ld16<0>(xr[4], xb, o2); ld16<64>(xr[5], xb, o2); ld16<128>(xr[6], xb, o2); ld16<192>(xr[7], xb, o2);
    }
    if (threadIdx.x < 16) lds_st(lds0 + L_FULL + threadIdx.x * 4, 0u);
    const unsigned call = (unsigned)__builtin_amdgcn_readfirstlane(ld4_sys_wait(myws, OFF_CNT + b * 128)) + 1u;
    asm volatile("s_waitcnt lgkmcnt(0)\n\ts_barrier" ::: "memory");
    const unsigned xs_off = OFF_XS + (call & 1u) * 4 * MiB;

    if (wave >= 4) {
#pragma unroll
        for (int i = 0; i < 8; ++i) asm volatile("" : "+v"(xr[i]));
        u32x4 h[4];
#pragma unroll
        for (int i = 0; i < 4; ++i) {
            const f32x4 lo = xr[2 * i], hi = xr[2 * i + 1];
            h[i] = (u32x4){pk(lo.x, lo.y), pk(lo.z, lo.w), pk(hi.x, hi.y), pk(hi.z, hi.w)};
        }
        const unsigned o = xs_off + (unsigned)(((b & 3) * NST + (b >> 2)) * A_TILE + (wave - 4) * 4096 + lane * 16);
        st16_sys<0>(myws, o, h[0]); st16_sys<1024>(myws, o, h[1]); st16_sys<2048>(myws, o, h[2]); st16_sys<3072>(myws, o, h[3]);
        if (wave == 4 && lane == 0) st4_sys(myws, OFF_CNT + b * 128, call);
        waitvm<0>();
        unsigned n = 0;
        if (lane == 0) asm volatile("ds_add_rtn_u32 %0, %1, %2\n\ts_waitcnt lgkmcnt(0)" : "=&v"(n) : "v"(lds0 + L_XCNT), "v"(1u) : "memory");
        n = __builtin_amdgcn_readfirstlane(n);
        if (n == 3u && lane < NR) {
            char* fp = (lane == rank) ? (char*)myws + OFF_AFLAG + (rank * 256 + b) * 64
                                      : P.ws[lane] + OFF_XFLAG + (rank * 256 + b) * 4;
            __hip_atomic_store((unsigned*)fp, call, __ATOMIC_RELAXED, __HIP_MEMORY_SCOPE_SYSTEM);
        }
        if (wave == 4) STAMP(1);
    } else {
        const int wcb = b & 7, pr = b >> 3;
        const float* wp = P.w + (size_t)(pr * 128 + q * 4) * N_ALL + rank * N_LOC + wcb * 256 + wave * 64 + c * 4;
        f32x4 wr[32];
#pragma unroll
        for (int i = 0; i < 32; ++i)
            wr[i] = *(const f32x4*)(wp + (size_t)((i >> 2) * 16 + (i & 3)) * N_ALL);
#pragma unroll
        for (int t = 0; t < 4; ++t) {
            const unsigned o = OFF_WB + (unsigned)((wcb * NST + pr * 2 + (t >> 1)) * B_TILE + wave * 8192 + (t & 1) * 4096 + lane * 16);
            const f32x4* r = wr + t * 8;
            st16_sys<0>(myws, o, (u32x4){pk(r[0].x, r[1].x), pk(r[2].x, r[3].x), pk(r[4].x, r[5].x), pk(r[6].x, r[7].x)});
            st16_sys<1024>(myws, o, (u32x4){pk(r[0].y, r[1].y), pk(r[2].y, r[3].y), pk(r[4].y, r[5].y), pk(r[6].y, r[7].y)});
            st16_sys<2048>(myws, o, (u32x4){pk(r[0].z, r[1].z), pk(r[2].z, r[3].z), pk(r[4].z, r[5].z), pk(r[6].z, r[7].z)});
            st16_sys<3072>(myws, o, (u32x4){pk(r[0].w, r[1].w), pk(r[2].w, r[3].w), pk(r[4].w, r[5].w), pk(r[6].w, r[7].w)});
        }
        waitvm<0>();
        if (lane == 0) st4_sys(myws, OFF_WFLAG + (wcb * 32 + pr) * 64 + wave * 4, call);
        if (wave == 0) STAMP(2);
    }

    if (wave == 7) {
        const int pi = idx >> 2;
        if ((idx & 3) != 0 || pi >= NR - 1) return;
        const int p = (rank + 1 + pi) & (NR - 1);
        const char* sb = uni(P.ws[p] + xs_off);
        const char* db = uni(myws + OFF_XA + p * 4 * MiB);
        poll_all(myws, OFF_XFLAG + (unsigned)(p * 256 + (lane & 31) * 8 + grp) * 4, call);
        STAMP(3);
        constexpr int WIN = 8;
        u32x4 reg[WIN] = {};
        auto toff = [&](int j) -> unsigned {
            const int t = j * 8 + grp;
            return (unsigned)(((t & 3) * NST + (t >> 2)) * A_TILE + lane * 16);
        };
        unsigned ro = toff(0);
        ld16_sys<0>(reg[0], sb, ro); ld16_sys<1024>(reg[1], sb, ro); ld16_sys<2048>(reg[2], sb, ro); ld16_sys<3072>(reg[3], sb, ro);
        ld16_sys<0>(reg[4], sb, ro + 4096); ld16_sys<1024>(reg[5], sb, ro + 4096); ld16_sys<2048>(reg[6], sb, ro + 4096); ld16_sys<3072>(reg[7], sb, ro + 4096);
        waitvm<0>();
        for (int h = 0; h < 63; ++h) {
            const unsigned no = toff((h + 1) >> 1) + ((h + 1) & 1) * 8192;
#define PSTEP(i) do { waitvm<2 * (WIN - 1)>(); asm volatile("" : "+v"(reg[i])); \
            st16_sys<((i) & 3) * 1024>(db, ro + ((i) >> 2) * 4096, reg[i]); \
            ld16_sys<((i) & 3) * 1024>(reg[i], sb, no + ((i) >> 2) * 4096); } while (0)
            PSTEP(0); PSTEP(1); PSTEP(2); PSTEP(3); PSTEP(4); PSTEP(5); PSTEP(6); PSTEP(7);
            if ((h & 1) == 0 && h > 0 && lane == 0)
                st4_sys(myws, OFF_AFLAG + (unsigned)(p * 256 + ((h >> 1) - 1) * 8 + grp) * 64, call);
            ro = no;
        }
        waitvm<0>();
#pragma unroll
        for (int i = 0; i < WIN; ++i) asm volatile("" : "+v"(reg[i]));
        st16_sys<0>(db, ro, reg[0]); st16_sys<1024>(db, ro, reg[1]); st16_sys<2048>(db, ro, reg[2]); st16_sys<3072>(db, ro, reg[3]);
        st16_sys<0>(db, ro + 4096, reg[4]); st16_sys<1024>(db, ro + 4096, reg[5]); st16_sys<2048>(db, ro + 4096, reg[6]); st16_sys<3072>(db, ro + 4096, reg[7]);
        waitvm<0>();
        if (lane == 0) st4_sys(myws, OFF_AFLAG + (unsigned)(p * 256 + 31 * 8 + grp) * 64, call);
        STAMP(4);
        return;
    }

    if (wave >= 4) {
        const int j = wave - 4;
        const char* ab = uni(myws + (src == rank ? xs_off : OFF_XA + src * 4 * MiB) + rb * NST * A_TILE);
        const char* bb = uni(myws + OFF_WB + cb * NST * B_TILE);
        const unsigned slot = lds0 + j * SLOT;
        const unsigned v0 = lane * 16;
        for (int s = j; s < NST; s += NSLOT) {
            const unsigned fo = lane < 4 ? OFF_WFLAG + (cb * 32 + (s >> 1)) * 64 + lane * 4
                                         : OFF_AFLAG + (src * 256 + s * 4 + rb) * 64;
            poll_all(myws, fo, call);
            if (s >= NSLOT) {
                while (!__all(ge(lds_ld_wait(lds0 + L_FREE + (lane & 3) * 4), (unsigned)(s - NSLOT + 1)))) __builtin_amdgcn_s_sleep(1);
            }
            const char* as = ab + s * A_TILE;
            const char* bs = bb + s * B_TILE;
#pragma unroll
            for (int g = 0; g < 4; ++g) glds4k(as, v0 + g * 4096, slot + g * 4096);
#pragma unroll
            for (int g = 0; g < 8; ++g) glds4k(bs, v0 + g * 4096, slot + A_TILE + g * 4096);
            const unsigned tmo = ld4_sys_wait(myws, OFF_TMO);
            if (tmo == call) lds_st(lds0 + L_ABORT, 1u);
            lds_st(lds0 + L_FULL + j * 4, (unsigned)(s + 1));
        }
        return;
    }

    f32x4 acc[8][4];
#pragma unroll
    for (int m = 0; m < 8; ++m)
#pragma unroll
        for (int f = 0; f < 4; ++f) acc[m][f] = (f32x4){0.f, 0.f, 0.f, 0.f};
    f16x8 A0[8], B0[4], A1[8], B1[4];
    const LAS char* la = (const LAS char*)smem + lane * 16;
    const LAS char* lb = la + A_TILE + wave * 8192;
#define LOADF(A, B, so, t) do { \
        _Pragma("unroll") for (int f = 0; f < 4; ++f) B[f] = *(const LAS f16x8*)(lb + (so) + (t) * 4096 + f * 1024); \
        _Pragma("unroll") for (int m = 0; m < 8; ++m) A[m] = *(const LAS f16x8*)(la + (so) + (m * 2 + (t)) * 1024); } while (0)
#define MMA(A, B) do { \
        _Pragma("unroll") for (int m = 0; m < 8; ++m) \
        _Pragma("unroll") for (int f = 0; f < 4; ++f) \
            acc[m][f] = __builtin_amdgcn_mfma_f32_16x16x32_f16(A[m], B[f], acc[m][f], 0, 0, 0); } while (0)
#define INTERLEAVE() do { \
        _Pragma("unroll") for (int i = 0; i < 12; ++i) { \
            __builtin_amdgcn_sched_group_barrier(0x008, 2, 0); __builtin_amdgcn_sched_group_barrier(0x100, 1, 0); } \
        __builtin_amdgcn_sched_group_barrier(0x008, 8, 0); } while (0)

    while (!ge(__builtin_amdgcn_readfirstlane(lds_ld_wait(lds0 + L_FULL)), 1u)) __builtin_amdgcn_s_sleep(1);
    if (wave == 0) STAMP(7);
    LOADF(A0, B0, 0, 0);
    unsigned so = 0;
    int sl = 0;
    for (int s = 0; s < NST - 1; ++s) {
        const int sn = sl == NSLOT - 1 ? 0 : sl + 1;
        const volatile LAS unsigned* fullp = (const volatile LAS unsigned*)((LAS char*)smem + L_FULL) + sn;
        unsigned fv = *fullp;
        LOADF(A1, B1, so, 1);
        MMA(A0, B0);
        INTERLEAVE();
        __builtin_amdgcn_sched_barrier(0);
        lds_st(lds0 + L_FREE + wave * 4, (unsigned)(s + 1));
        while (!ge(__builtin_amdgcn_readfirstlane(fv), (unsigned)(s + 2))) fv = *fullp;
        asm volatile("" ::: "memory");
        sl = sn; so = sn * SLOT;
        LOADF(A0, B0, so, 0);
        MMA(A1, B1);
        INTERLEAVE();
        __builtin_amdgcn_sched_barrier(0);
    }
    LOADF(A1, B1, so, 1);
    MMA(A0, B0);
    INTERLEAVE();
    __builtin_amdgcn_sched_barrier(0);
    MMA(A1, B1);
    if (wave == 0) STAMP(8);

    const bool bad = __builtin_amdgcn_readfirstlane(lds_ld_wait(lds0 + L_ABORT)) != 0u;
    float* op = P.out + (size_t)(src * M_LOC + rb * 128 + q * 4) * N_LOC + cb * 256 + wave * 64 + c * 4;
    const float nanv = __builtin_nanf("");
#pragma unroll
    for (int m = 0; m < 8; ++m)
#pragma unroll
        for (int r = 0; r < 4; ++r) {
            f32x4 v = {acc[m][0][r], acc[m][1][r], acc[m][2][r], acc[m][3][r]};
            if (bad) v = (f32x4){nanv, nanv, nanv, nanv};
            *(f32x4*)(op + (size_t)(m * 16 + r) * N_LOC) = v;
        }
}

}

extern "C" void kernel_launch(int rank, int nranks, ncclComm_t comm,
                              hipStream_t stream,
                              void* const* d_in, const int* in_sizes, int n_in,
                              void* d_out, int out_size,
                              const int* in_dtypes, int out_dtype,
                              void* const* d_out_all,
                              void* const* d_ws_all, size_t ws_bytes) {
    (void)comm; (void)in_dtypes; (void)out_dtype; (void)d_out_all;
    if (nranks != NR || (unsigned)rank >= (unsigned)NR || n_in != 2 || in_sizes[0] != M_LOC * KDIM ||
        in_sizes[1] != KDIM * N_ALL || out_size != NR * M_LOC * N_LOC || ws_bytes < WS_NEED) {
        fprintf(stderr, "kernel_launch: unexpected shapes\n");
        return;
    }
    HIP_CHECK(hipFuncSetAttribute((const void*)gemm_a2a_kernel, hipFuncAttributeMaxDynamicSharedMemorySize, (int)LDS_BYTES));
    Params P;
    P.x = (const float*)d_in[0];
    P.w = (const float*)d_in[1];
    P.out = (float*)d_out;
    for (int p = 0; p < NR; ++p) P.ws[p] = (char*)d_ws_all[p];
    P.rank = rank;
    P.pad = 0;
    hipLaunchKernelGGL(gemm_a2a_kernel, dim3(256), dim3(512), LDS_BYTES, stream, P);
    HIP_CHECK(hipGetLastError());
}
```

```cpp
#include <hip/hip_runtime.h>
#include <rccl/rccl.h>
#include <cstdio>

#define HIP_CHECK(expr) do { hipError_t _e = (expr); if (_e != hipSuccess) { \
    fprintf(stderr, "HIP error %s at %s:%d\n", hipGetErrorString(_e), __FILE__, __LINE__); \
    return; } } while (0)

namespace {

constexpr int NR = 8;
constexpr int KDIM = 4096, N_ALL = 16384, N_LOC = 2048, M_LOC = 512;
constexpr int NST = 64;
constexpr unsigned A_TILE = 16384;
constexpr unsigned B_TILE = 32768;
constexpr unsigned A_LDS = A_TILE + 2048;
constexpr unsigned SLOT = A_LDS + B_TILE;
constexpr int NSLOT = 3;
constexpr unsigned MiB = 1u << 20;

constexpr unsigned OFF_CNT = 0;
constexpr unsigned OFF_TMO = 32 * 1024;
constexpr unsigned OFF_XFLAG = 64 * 1024;
constexpr unsigned OFF_AFLAG = 128 * 1024;
constexpr unsigned OFF_WFLAG = 256 * 1024;
constexpr unsigned OFF_DBG = 512 * 1024;
constexpr unsigned OFF_XS = 1 * MiB;
constexpr unsigned OFF_XA = 9 * MiB;
constexpr unsigned OFF_WB = 41 * MiB;
constexpr size_t WS_NEED = 57ull * MiB;

constexpr unsigned L_FULL = NSLOT * SLOT;
constexpr unsigned L_FREE = L_FULL + 16;
constexpr unsigned L_XCNT = L_FREE + 16;
constexpr unsigned L_ABORT = L_XCNT + 4;
constexpr unsigned LDS_BYTES = L_FULL + 64;

constexpr unsigned long long DEADLINE_TICKS = 50000000ull;

typedef unsigned u32x4 __attribute__((ext_vector_type(4)));
typedef float f32x4 __attribute__((ext_vector_type(4)));
typedef float f32x2 __attribute__((ext_vector_type(2)));
typedef _Float16 f16x2 __attribute__((ext_vector_type(2)));
typedef _Float16 f16x8 __attribute__((ext_vector_type(8)));
#define LAS __attribute__((address_space(3)))

struct Params {
    const float* x;
    const float* w;
    float* out;
    char* ws[NR];
    int rank;
    int pad;
};

__device__ __forceinline__ unsigned pk(float a, float b) {
    f32x2 v = {a, b};
    return __builtin_bit_cast(unsigned, __builtin_convertvector(v, f16x2));
}
template <int N> __device__ __forceinline__ void waitvm() { asm volatile("s_waitcnt vmcnt(%0)" :: "i"(N) : "memory"); }
__device__ __forceinline__ const char* uni(const char* p) {
    unsigned long long v = (unsigned long long)p;
    unsigned lo = __builtin_amdgcn_readfirstlane((unsigned)v), hi = __builtin_amdgcn_readfirstlane((unsigned)(v >> 32));
    return (const char*)(((unsigned long long)hi << 32) | lo);
}
template <int IMM> __device__ __forceinline__ void ld16(f32x4& d, const char* base, unsigned voff) {
    asm volatile("global_load_dwordx4 %0, %1, %2 offset:%3" : "=v"(d) : "v"(voff), "s"(base), "i"(IMM) : "memory");
}
template <int IMM> __device__ __forceinline__ void ld16_sys(u32x4& d, const char* base, unsigned voff) {
    asm volatile("global_load_dwordx4 %0, %1, %2 offset:%3 sc0 sc1" : "+v"(d) : "v"(voff), "s"(base), "i"(IMM) : "memory");
}
template <int IMM> __device__ __forceinline__ void st16_sys(const char* base, unsigned voff, u32x4 v) {
    asm volatile("global_store_dwordx4 %0, %1, %2 offset:%3 sc0 sc1\n\ts_nop 1" :: "v"(voff), "v"(v), "s"(base), "i"(IMM) : "memory");
}
__device__ __forceinline__ void st4_sys(const char* base, unsigned voff, unsigned v) {
    asm volatile("global_store_dword %0, %1, %2 sc0 sc1" :: "v"(voff), "v"(v), "s"(base) : "memory");
}
__device__ __forceinline__ unsigned ld4_sys_wait(const char* base, unsigned voff) {
    unsigned v;
    asm volatile("global_load_dword %0, %1, %2 sc0 sc1\n\ts_waitcnt vmcnt(0)" : "=&v"(v) : "v"(voff), "s"(base) : "memory");
    return v;
}
__device__ __forceinline__ void glds4k(const char* base, unsigned voff, unsigned lds) {
    unsigned keep;
    asm volatile("s_mov_b32 %0, m0\n\ts_mov_b32 m0, %3\n\ts_nop 0\n\t"
                 "global_load_lds_dwordx4 %1, %2 sc0 sc1\n\t"
                 "global_load_lds_dwordx4 %1, %2 offset:1024 sc0 sc1\n\t"
                 "global_load_lds_dwordx4 %1, %2 offset:2048 sc0 sc1\n\t"
                 "global_load_lds_dwordx4 %1, %2 offset:3072 sc0 sc1\n\t"
                 "s_mov_b32 m0, %0"
                 : "=&s"(keep) : "v"(voff), "s"(base), "s"(lds) : "memory");
}
__device__ __forceinline__ void glds2k(const char* base, unsigned voff, unsigned lds) {
    unsigned keep;
    asm volatile("s_mov_b32 %0, m0\n\ts_mov_b32 m0, %3\n\ts_nop 0\n\t"
                 "global_load_lds_dwordx4 %1, %2 sc0 sc1\n\t"
                 "global_load_lds_dwordx4 %1, %2 offset:1024 sc0 sc1\n\t"
                 "s_mov_b32 m0, %0"
                 : "=&s"(keep) : "v"(voff), "s"(base), "s"(lds) : "memory");
}
__device__ __forceinline__ void lds_st(unsigned addr, unsigned v) {
    asm volatile("ds_write_b32 %0, %1" :: "v"(addr), "v"(v) : "memory");
}
__device__ __forceinline__ unsigned lds_ld_wait(unsigned addr) {
    unsigned v;
    asm volatile("ds_read_b32 %0, %1\n\ts_waitcnt lgkmcnt(0)" : "=&v"(v) : "v"(addr) : "memory");
    return v;
}
__device__ __forceinline__ bool ge(unsigned v, unsigned want) { return (int)(v - want) >= 0; }

__device__ __forceinline__ void poll_all(const char* myws, unsigned voff, unsigned call) {
    if (__all(ge(ld4_sys_wait(myws, voff), call))) return;
    const unsigned long long t0 = __builtin_amdgcn_s_memrealtime();
    for (;;) {
        if (__all(ge(ld4_sys_wait(myws, voff), call))) return;
        if (__builtin_amdgcn_s_memrealtime() - t0 > 2000) {
            if ((unsigned)__builtin_amdgcn_readfirstlane(ld4_sys_wait(myws, OFF_TMO)) == call) return;
            if (__builtin_amdgcn_s_memrealtime() - t0 > DEADLINE_TICKS) {
                st4_sys(myws, OFF_TMO, call);
                waitvm<0>();
                return;
            }
        }
        __builtin_amdgcn_s_sleep(1);
    }
}

__device__ __forceinline__ unsigned long long poll_bit(const char* myws, unsigned voff, unsigned call, int j) {
    const unsigned long long t0 = __builtin_amdgcn_s_memrealtime();
    for (;;) {
        const unsigned long long m = __ballot(ge(ld4_sys_wait(myws, voff), call));
        if ((m >> j) & 1) return m;
        if (__builtin_amdgcn_s_memrealtime() - t0 > 2000) {
            if ((unsigned)__builtin_amdgcn_readfirstlane(ld4_sys_wait(myws, OFF_TMO)) == call) return ~0ull;
            if (__builtin_amdgcn_s_memrealtime() - t0 > DEADLINE_TICKS) {
                st4_sys(myws, OFF_TMO, call);
                waitvm<0>();
                return ~0ull;
            }
        }
    }
}

#define STAMP(i) do {} while (0)

__device__ __forceinline__ void convert_w(const Params& P, const char* myws, int u, int lane, unsigned call) {
    const int st = u >> 5, cg = u & 31, q = lane >> 4, c = lane & 15;
    const float* wp = P.w + (size_t)(st * 64 + q * 4) * N_ALL + P.rank * N_LOC + cg * 64 + c * 4;
    f32x4 wr[16];
#pragma unroll
    for (int i = 0; i < 16; ++i)
        wr[i] = *(const f32x4*)(wp + (size_t)((i >> 2) * 16 + (i & 3)) * N_ALL);
#pragma unroll
    for (int t = 0; t < 2; ++t) {
        const unsigned o = OFF_WB + (unsigned)(((cg >> 2) * NST + st) * B_TILE + (cg & 3) * 8192 + t * 4096 + lane * 16);
        const f32x4* r = wr + t * 8;
        st16_sys<0>(myws, o, (u32x4){pk(r[0].x, r[1].x), pk(r[2].x, r[3].x), pk(r[4].x, r[5].x), pk(r[6].x, r[7].x)});
        st16_sys<1024>(myws, o, (u32x4){pk(r[0].y, r[1].y), pk(r[2].y, r[3].y), pk(r[4].y, r[5].y), pk(r[6].y, r[7].y)});
        st16_sys<2048>(myws, o, (u32x4){pk(r[0].z, r[1].z), pk(r[2].z, r[3].z), pk(r[4].z, r[5].z), pk(r[6].z, r[7].z)});
        st16_sys<3072>(myws, o, (u32x4){pk(r[0].w, r[1].w), pk(r[2].w, r[3].w), pk(r[4].w, r[5].w), pk(r[6].w, r[7].w)});
    }
    waitvm<0>();
    if (lane == 0) st4_sys(myws, OFF_WFLAG + (unsigned)(((cg >> 2) * NST + st) * 64 + (cg & 3) * 4), call);
}

template <int MT>
__device__ __forceinline__ void consume(const Params& P, char* smem, const char* myws, int b, int wave, int lane,
                                        int row0, int rowl, int cb) {
    const unsigned lds0 = (unsigned)(unsigned long long)(LAS char*)smem;
    f32x4 acc[MT][4];
#pragma unroll
    for (int m = 0; m < MT; ++m)
#pragma unroll
        for (int f = 0; f < 4; ++f) acc[m][f] = (f32x4){0.f, 0.f, 0.f, 0.f};
    f16x8 A[MT], B0[4], B1[4];
    const LAS char* la = (const LAS char*)smem + lane * 16;
    const LAS char* lb = la + A_LDS + wave * 8192;
#define DSON true
#define KSTEP(BC, BN, so, t, LOAD, MID) do { \
        _Pragma("unroll") for (int m = 0; m < MT; ++m) { \
            acc[m][0] = __builtin_amdgcn_mfma_f32_16x16x32_f16(A[m], BC[0], acc[m][0], 0, 0, 0); \
            acc[m][1] = __builtin_amdgcn_mfma_f32_16x16x32_f16(A[m], BC[1], acc[m][1], 0, 0, 0); \
            __builtin_amdgcn_sched_barrier(0); \
            if (DSON && LOAD && m < 4) BN[m] = *(const LAS f16x8*)(lb + (so) + (t) * 4096 + m * 1024); \
            if (m == (MT > 4 ? 4 : 0)) { MID; } \
            __builtin_amdgcn_sched_barrier(0); \
            acc[m][2] = __builtin_amdgcn_mfma_f32_16x16x32_f16(A[m], BC[2], acc[m][2], 0, 0, 0); \
            acc[m][3] = __builtin_amdgcn_mfma_f32_16x16x32_f16(A[m], BC[3], acc[m][3], 0, 0, 0); \
            __builtin_amdgcn_sched_barrier(0); \
            if (DSON && LOAD) A[m] = *(const LAS f16x8*)(la + (so) + (m * 2 + (t)) * 1024); \
            __builtin_amdgcn_sched_barrier(0); \
        } } while (0)

    while (!ge(__builtin_amdgcn_readfirstlane(lds_ld_wait(lds0 + L_FULL)), 1u)) __builtin_amdgcn_s_sleep(1);
    if (wave == 0) STAMP(7);
    __builtin_amdgcn_s_waitcnt(0xC07F);
#pragma unroll
    for (int f = 0; f < 4; ++f) B0[f] = *(const LAS f16x8*)(lb + f * 1024);
#pragma unroll
    for (int m = 0; m < MT; ++m) A[m] = *(const LAS f16x8*)(la + m * 2048);
    unsigned so = 0;
    int sl = 0;
    for (int s = 0; s < NST - 1; ++s) {
        const int sn = sl == NSLOT - 1 ? 0 : sl + 1;
        const volatile LAS unsigned* fullp = (const volatile LAS unsigned*)((LAS char*)smem + L_FULL) + sn;
        unsigned fv;
        KSTEP(B0, B1, so, 1, true, fv = *fullp);
        *((volatile LAS unsigned*)((LAS char*)smem + L_FREE) + wave) = (unsigned)(s + 1);
#pragma unroll
        for (int m = 0; m < MT; ++m) asm volatile("" : "+v"(acc[m][0]), "+v"(acc[m][1]), "+v"(acc[m][2]), "+v"(acc[m][3]));
        if (__builtin_expect(!ge(__builtin_amdgcn_readfirstlane(fv), (unsigned)(s + 2)), 0)) {
            while (!ge(__builtin_amdgcn_readfirstlane(lds_ld_wait(lds0 + L_FULL + sn * 4)), (unsigned)(s + 2))) __builtin_amdgcn_s_sleep(1);
        }
        asm volatile("" ::: "memory");
        sl = sn; so = sn * SLOT;
        KSTEP(B1, B0, so, 0, true, (void)0);
    }
    KSTEP(B0, B1, so, 1, true, (void)0);
    KSTEP(B1, B0, so, 0, false, (void)0);
    if (wave == 0) STAMP(8);

    const bool bad = __builtin_amdgcn_readfirstlane(lds_ld_wait(lds0 + L_ABORT)) != 0u;
    float* op = P.out + (size_t)((lane >> 4) * 4) * N_LOC + cb * 256 + wave * 64 + (lane & 15) * 4;
    const float nanv = __builtin_nanf("");
#pragma unroll
    for (int m = 0; m < MT; ++m) {
        const int row = m < MT - 1 ? row0 + m * 16 : rowl;
#pragma unroll
        for (int r = 0; r < 4; ++r) {
            f32x4 v = {acc[m][0][r], acc[m][1][r], acc[m][2][r], acc[m][3][r]};
            if (bad) v = (f32x4){nanv, nanv, nanv, nanv};
            *(f32x4*)(op + (size_t)(row + r) * N_LOC) = v;
        }
    }
#undef KSTEP
}

constexpr int NPULLCU = 3;
#define NPW 4
constexpr int NGEMM = 8 * (32 - NPULLCU);
constexpr int W_FIRST = NGEMM * 4;

__global__ __launch_bounds__(512, 2) void gemm_a2a_kernel(Params P) {
    extern __shared__ __attribute__((aligned(16))) char smem[];
    const int lane = threadIdx.x & 63;
    const int wave = __builtin_amdgcn_readfirstlane(threadIdx.x >> 6);
    const int b = blockIdx.x;
    const int rank = P.rank;
    const char* myws = uni(P.ws[rank]);
    const unsigned lds0 = (unsigned)(unsigned long long)(LAS char*)smem;
    const int grp = b & 7, idx = b >> 3;

    if (wave == 0) STAMP(0);
    f32x4 xr[8];
    if (wave >= 4) {
        const unsigned o = (unsigned)((((b & 3) * 128 + (wave - 4) * 32 + (lane & 15)) * KDIM + (b >> 2) * 64 + (lane >> 4) * 4) * 4);
        const char* xb = uni((const char*)P.x);
        ld16<0>(xr[0], xb, o); ld16<64>(xr[1], xb, o); ld16<128>(xr[2], xb, o); ld16<192>(xr[3], xb, o);
        const unsigned o2 = o + 16 * KDIM * 4;
        ld16<0>(xr[4], xb, o2); ld16<64>(xr[5], xb, o2); ld16<128>(xr[6], xb, o2); ld16<192>(xr[7], xb, o2);
    }
    if (threadIdx.x < 16) lds_st(lds0 + L_FULL + threadIdx.x * 4, 0u);
    const unsigned call = (unsigned)__builtin_amdgcn_readfirstlane(ld4_sys_wait(myws, OFF_CNT + b * 128)) + 1u;
    const unsigned xs_off = OFF_XS + (call & 1u) * 4 * MiB;
    if (wave >= 4) {
        if (wave == 4) STAMP(13);
#pragma unroll
        for (int i = 0; i < 8; ++i) asm volatile("" : "+v"(xr[i]));
        u32x4 h[4];
#pragma unroll
        for (int i = 0; i < 4; ++i) {
            const f32x4 lo = xr[2 * i], hi = xr[2 * i + 1];
            h[i] = (u32x4){pk(lo.x, lo.y), pk(lo.z, lo.w), pk(hi.x, hi.y), pk(hi.z, hi.w)};
        }
        const unsigned o = xs_off + (unsigned)(((b & 3) * NST + (b >> 2)) * A_TILE + (wave - 4) * 4096 + lane * 16);
        st16_sys<0>(myws, o, h[0]); st16_sys<1024>(myws, o, h[1]); st16_sys<2048>(myws, o, h[2]); st16_sys<3072>(myws, o, h[3]);
        waitvm<0>();
    }
    asm volatile("s_waitcnt lgkmcnt(0)\n\ts_barrier" ::: "memory");
    if (wave == 4) {
        if (lane < NR) {
            char* fp = (lane == rank) ? (char*)myws + OFF_AFLAG + (rank * 256 + b) * 64
                                      : P.ws[lane] + OFF_XFLAG + (rank * 256 + b) * 4;
            __hip_atomic_store((unsigned*)fp, call, __ATOMIC_RELAXED, __HIP_MEMORY_SCOPE_SYSTEM);
        }
        if (lane == 0) st4_sys(myws, OFF_CNT + b * 128, call);
        STAMP(1);
    }

    if (idx < NPULLCU) {
        constexpr int PL = 8 * NPULLCU * 4 / 7;
        const int pw = (idx * 8 + grp) * 4 + wave;
        if (wave >= 4 || pw >= 7 * PL) return;
        const int p = (rank + 1 + pw % 7) & (NR - 1), li = pw / 7;
        const char* sb = uni(P.ws[p] + xs_off);
        const char* db = uni(myws + OFF_XA + p * 4 * MiB);
        const unsigned xfo = OFF_XFLAG + (unsigned)((p * 256 + ((li + lane * PL < 512 ? li + lane * PL : li) >> 1)) * 4);
        unsigned long long rdy = poll_bit(myws, xfo, call, 0);
        if (wave == 0) STAMP(3);
        u32x4 ra[4] = {}, rb4[4] = {};
        auto toff = [&](int hu) -> unsigned {
            const int t = hu >> 1;
            return (unsigned)(((t & 3) * NST + (t >> 2)) * A_TILE + (hu & 1) * 8192 + lane * 16);
        };
        auto load4 = [&](u32x4 (&r)[4], unsigned o) {
            ld16_sys<0>(r[0], sb, o); ld16_sys<1024>(r[1], sb, o); ld16_sys<2048>(r[2], sb, o); ld16_sys<3072>(r[3], sb, o); };
        auto store4 = [&](u32x4 (&r)[4], unsigned o) {
            asm volatile("" : "+v"(r[0]), "+v"(r[1]), "+v"(r[2]), "+v"(r[3]));
            st16_sys<0>(db, o, r[0]); st16_sys<1024>(db, o, r[1]); st16_sys<2048>(db, o, r[2]); st16_sys<3072>(db, o, r[3]); };
        auto flag = [&](int hu) {
            if (lane == 0) st4_sys(myws, OFF_AFLAG + (unsigned)(hu < 0 ? (p * 256 + li) * 64 + 32 : (p * 256 + (hu >> 1)) * 64 + (hu & 1) * 4), call); };
        unsigned o = toff(li);
        load4(ra, o);
        waitvm<0>();
        int j = 0, hu = li;
        for (;; hu += PL, ++j) {
            waitvm<4>();
            load4(rb4, o + 4096);
            waitvm<4>();
            flag(hu - PL);
            store4(ra, o);
            if (hu + PL >= 512) break;
            if (!((rdy >> (j + 1)) & 1)) rdy = poll_bit(myws, xfo, call, j + 1);
            const unsigned on = toff(hu + PL);
            waitvm<5>();
            load4(ra, on);
            store4(rb4, o + 4096);
            o = on;
        }
        waitvm<4>();
        store4(rb4, o + 4096);
        waitvm<0>();
        flag(hu);
        if (wave == 0) STAMP(4);
        return;
    }

    const int gi = (idx - NPULLCU) * 8 + grp;
    const int ti = idx - NPULLCU;
    const bool small = ti == 28;
    const int rb = small ? 3 : grp >> 1;
    const int src = small ? rank : (rank + 1 + (ti >> 2)) & (NR - 1);
    const int cb = (grp & 1) * 4 + (small ? grp >> 1 : ti & 3);
    const int lg = small ? 31 : (ti >> 2) * 4 + rb;

    if (wave == 7) {
        for (int j = 0; W_FIRST + j * NGEMM + gi < 2048; ++j) {
            while (!ge(__builtin_amdgcn_readfirstlane(lds_ld_wait(lds0 + L_FREE)), (unsigned)(4 * j + 2))) __builtin_amdgcn_s_sleep(8);
            convert_w(P, myws, W_FIRST + j * NGEMM + gi, lane, call);
        }
        STAMP(14);
        return;
    }

    if (wave >= 4) {
        const int j = wave - 4;
        const char* ab = uni(myws + (small ? xs_off + 8192 : OFF_XA + src * 4 * MiB) + rb * NST * A_TILE);
        const char* lb = uni(myws + xs_off + (lg >> 3) * NST * A_TILE + (lg & 7) * 2048);
        const char* bb = uni(myws + OFF_WB + cb * NST * B_TILE);
        const unsigned slot = lds0 + j * SLOT;
        const unsigned v0 = lane * 16;
        for (int s = j; s < NST; s += NSLOT) {
            const unsigned fo = lane < 4 ? OFF_WFLAG + (cb * NST + s) * 64 + lane * 4
                              : (lane < 6 && !small) ? OFF_AFLAG + (src * 256 + s * 4 + rb) * 64 + (lane & 1) * 4
                                                     : OFF_AFLAG + (rank * 256 + s * 4 + (lg >> 3)) * 64;
            poll_all(myws, fo, call);
            if (s >= NSLOT) {
                while (!__all(ge(lds_ld_wait(lds0 + L_FREE + (lane & 3) * 4), (unsigned)(s - NSLOT + 1)))) __builtin_amdgcn_s_sleep(1);
            }
            const char* as = ab + s * A_TILE;
            const char* bs = bb + s * B_TILE;
            glds4k(as, v0, slot); glds4k(as, v0 + 4096, slot + 4096);
            if (!small) {
                glds4k(as, v0 + 8192, slot + 8192); glds4k(as, v0 + 12288, slot + 12288);
                glds2k(lb + s * A_TILE, v0, slot + A_TILE);
            }
#pragma unroll
            for (int g = 0; g < 8; ++g) glds4k(bs, v0 + g * 4096, slot + A_LDS + g * 4096);
            const unsigned tmo = ld4_sys_wait(myws, OFF_TMO);
            if (tmo == call) lds_st(lds0 + L_ABORT, 1u);
            lds_st(lds0 + L_FULL + j * 4, (unsigned)(s + 1));
        }
        return;
    }

    convert_w(P, myws, gi * 4 + wave, lane, call);
    if (wave == 0) STAMP(2);
    if (small) consume<4>(P, smem, myws, b, wave, lane, rank * M_LOC + 28 * 16, rank * M_LOC + 31 * 16, cb);
    else consume<9>(P, smem, myws, b, wave, lane, src * M_LOC + rb * 128, rank * M_LOC + lg * 16, cb);
}

}

extern "C" void kernel_launch(int rank, int nranks, ncclComm_t comm,
                              hipStream_t stream,
                              void* const* d_in, const int* in_sizes, int n_in,
                              void* d_out, int out_size,
                              const int* in_dtypes, int out_dtype,
                              void* const* d_out_all,
                              void* const* d_ws_all, size_t ws_bytes) {
    (void)comm; (void)in_dtypes; (void)out_dtype; (void)d_out_all;
    if (nranks != NR || (unsigned)rank >= (unsigned)NR || n_in != 2 || in_sizes[0] != M_LOC * KDIM ||
        in_sizes[1] != KDIM * N_ALL || out_size != NR * M_LOC * N_LOC || ws_bytes < WS_NEED) {
        fprintf(stderr, "kernel_launch: unexpected shapes\n");
        return;
    }
    HIP_CHECK(hipFuncSetAttribute((const void*)gemm_a2a_kernel, hipFuncAttributeMaxDynamicSharedMemorySize, (int)LDS_BYTES));
    Params P;
    P.x = (const float*)d_in[0];
    P.w = (const float*)d_in[1];
    P.out = (float*)d_out;
    for (int p = 0; p < NR; ++p) P.ws[p] = (char*)d_ws_all[p];
    P.rank = rank;
    P.pad = 0;
    hipLaunchKernelGGL(gemm_a2a_kernel, dim3(256), dim3(512), LDS_BYTES, stream, P);
    HIP_CHECK(hipGetLastError());
}
```

```cpp
#include <hip/hip_runtime.h>
#include <rccl/rccl.h>
#include <cstdio>

#define HIP_CHECK(expr) do { hipError_t _e = (expr); if (_e != hipSuccess) { \
    fprintf(stderr, "HIP error %s at %s:%d\n", hipGetErrorString(_e), __FILE__, __LINE__); \
    return; } } while (0)

namespace {

constexpr int NR = 8;
constexpr int KDIM = 4096, N_ALL = 16384, N_LOC = 2048, M_LOC = 512;
constexpr int NST = 64;
constexpr unsigned A_TILE = 16384;
constexpr unsigned B_TILE = 32768;
constexpr unsigned A_LDS = A_TILE + 4096;
constexpr unsigned SLOT = A_LDS + B_TILE;
constexpr int NSLOT = 3;
constexpr unsigned MiB = 1u << 20;

constexpr unsigned OFF_CNT = 0;
constexpr unsigned OFF_TMO = 32 * 1024;
constexpr unsigned OFF_TICKET = 33 * 1024;
constexpr unsigned OFF_AGG = 36 * 1024;
constexpr unsigned OFF_XFLAG = 64 * 1024;
constexpr unsigned OFF_AFLAG = 128 * 1024;
constexpr unsigned OFF_WFLAG = 256 * 1024;
constexpr unsigned OFF_DBG = 512 * 1024;
constexpr unsigned OFF_XS = 1 * MiB;
constexpr unsigned OFF_XA = 9 * MiB;
constexpr unsigned OFF_WB = 41 * MiB;
constexpr unsigned OFF_XP = 57 * MiB;
constexpr unsigned OFF_XA0 = 65 * MiB;
constexpr size_t WS_NEED = 69ull * MiB;
constexpr unsigned P_SUB = 3584, P_EXC = 2 * P_SUB;
#define PUSH0 2
#define DIRECT_SP 4
#define RAW_SP 2

constexpr unsigned L_FULL = NSLOT * SLOT;
constexpr unsigned L_FREE = L_FULL + 16;
constexpr unsigned L_XCNT = L_FREE + 16;
constexpr unsigned L_ABORT = L_XCNT + 4;
constexpr unsigned L_LIST = L_FULL + 64;
constexpr unsigned LDS_BYTES = L_LIST + 1024;

constexpr unsigned long long DEADLINE_TICKS = 50000000ull;

typedef unsigned u32x4 __attribute__((ext_vector_type(4)));
typedef unsigned u32x2 __attribute__((ext_vector_type(2)));
typedef float f32x4 __attribute__((ext_vector_type(4)));
typedef float f32x2 __attribute__((ext_vector_type(2)));
typedef _Float16 f16x2 __attribute__((ext_vector_type(2)));
typedef _Float16 f16x8 __attribute__((ext_vector_type(8)));
#define LAS __attribute__((address_space(3)))

struct Params {
    const float* x;
    const float* w;
    float* out;
    int rank;
    int pad;
    char* ws[NR];
};

__device__ __forceinline__ unsigned pk(float a, float b) {
    f32x2 v = {a, b};
    return __builtin_bit_cast(unsigned, __builtin_convertvector(v, f16x2));
}
template <int N> __device__ __forceinline__ void waitvm() { asm volatile("s_waitcnt vmcnt(%0)" :: "i"(N) : "memory"); }
__device__ __forceinline__ const char* uni(const char* p) {
    unsigned long long v = (unsigned long long)p;
    unsigned lo = __builtin_amdgcn_readfirstlane((unsigned)v), hi = __builtin_amdgcn_readfirstlane((unsigned)(v >> 32));
    return (const char*)(((unsigned long long)hi << 32) | lo);
}
template <int IMM> __device__ __forceinline__ void ld16(f32x4& d, const char* base, unsigned voff) {
    asm volatile("global_load_dwordx4 %0, %1, %2 offset:%3" : "=v"(d) : "v"(voff), "s"(base), "i"(IMM) : "memory");
}
template <int IMM> __device__ __forceinline__ void ld16_sys(u32x4& d, const char* base, unsigned voff) {
    asm volatile("global_load_dwordx4 %0, %1, %2 offset:%3 sc0 sc1" : "+v"(d) : "v"(voff), "s"(base), "i"(IMM) : "memory");
}
template <int IMM> __device__ __forceinline__ void st16_sys(const char* base, unsigned voff, u32x4 v) {
    asm volatile("global_store_dwordx4 %0, %1, %2 offset:%3 sc0 sc1\n\ts_nop 1" :: "v"(voff), "v"(v), "s"(base), "i"(IMM) : "memory");
}
__device__ __forceinline__ void st4_sys(const char* base, unsigned voff, unsigned v) {
    asm volatile("global_store_dword %0, %1, %2 sc0 sc1" :: "v"(voff), "v"(v), "s"(base) : "memory");
}
__device__ __forceinline__ unsigned ld4_sys_wait(const char* base, unsigned voff) {
    unsigned v;
    asm volatile("global_load_dword %0, %1, %2 sc0 sc1\n\ts_waitcnt vmcnt(0)" : "=&v"(v) : "v"(voff), "s"(base) : "memory");
    return v;
}
__device__ __forceinline__ void glds4k(const char* base, unsigned voff, unsigned lds) {
    unsigned keep;
    asm volatile("s_mov_b32 %0, m0\n\ts_mov_b32 m0, %3\n\ts_nop 0\n\t"
                 "global_load_lds_dwordx4 %1, %2 sc0 sc1\n\t"
                 "global_load_lds_dwordx4 %1, %2 offset:1024 sc0 sc1\n\t"
                 "global_load_lds_dwordx4 %1, %2 offset:2048 sc0 sc1\n\t"
                 "global_load_lds_dwordx4 %1, %2 offset:3072 sc0 sc1\n\t"
                 "s_mov_b32 m0, %0"
                 : "=&s"(keep) : "v"(voff), "s"(base), "s"(lds) : "memory");
}
__device__ __forceinline__ void glds2k(const char* base, unsigned voff, unsigned lds) {
    unsigned keep;
    asm volatile("s_mov_b32 %0, m0\n\ts_mov_b32 m0, %3\n\ts_nop 0\n\t"
                 "global_load_lds_dwordx4 %1, %2 sc0 sc1\n\t"
                 "global_load_lds_dwordx4 %1, %2 offset:1024 sc0 sc1\n\t"
                 "s_mov_b32 m0, %0"
                 : "=&s"(keep) : "v"(voff), "s"(base), "s"(lds) : "memory");
}
__device__ __forceinline__ void lds_st(unsigned addr, unsigned v) {
    asm volatile("ds_write_b32 %0, %1" :: "v"(addr), "v"(v) : "memory");
}
__device__ __forceinline__ unsigned lds_ld_wait(unsigned addr) {
    unsigned v;
    asm volatile("ds_read_b32 %0, %1\n\ts_waitcnt lgkmcnt(0)" : "=&v"(v) : "v"(addr) : "memory");
    return v;
}
__device__ __forceinline__ bool ge(unsigned v, unsigned want) { return (int)(v - want) >= 0; }

#define POLL_SLEEP 2
__device__ __forceinline__ bool give_up(const char* myws, unsigned call, unsigned long long t0) {
    if (__builtin_amdgcn_s_memrealtime() - t0 <= 2000) return false;
    if ((unsigned)__builtin_amdgcn_readfirstlane(ld4_sys_wait(myws, OFF_TMO)) == call) return true;
    if (__builtin_amdgcn_s_memrealtime() - t0 <= DEADLINE_TICKS) return false;
    st4_sys(myws, OFF_TMO, call);
    waitvm<0>();
    return true;
}
__device__ __forceinline__ unsigned long long poll_mask(const char* myws, unsigned voff, unsigned call, unsigned long long need) {
    unsigned long long m = __ballot(ge(ld4_sys_wait(myws, voff), call));
    if ((m & need) == need) return m;
    const unsigned long long t0 = __builtin_amdgcn_s_memrealtime();
    for (;;) {
        m = __ballot(ge(ld4_sys_wait(myws, voff), call));
        if ((m & need) == need) return m;
        if (give_up(myws, call, t0)) return ~0ull;
        __builtin_amdgcn_s_sleep(POLL_SLEEP);
    }
}
__device__ __forceinline__ unsigned long long poll_units(const char* myws, unsigned voff, unsigned call, int j, unsigned& rawbits) {
    const unsigned long long t0 = __builtin_amdgcn_s_memrealtime();
    for (;;) {
        u32x4 v, w;
        asm volatile("global_load_dwordx4 %0, %2, %3 sc0 sc1\n\tglobal_load_dwordx4 %1, %2, %3 offset:16 sc0 sc1\n\ts_waitcnt vmcnt(0)"
                     : "=&v"(v), "=&v"(w) : "v"(voff), "s"(myws) : "memory");
        const unsigned c2 = 2u * call;
        const unsigned neg = (v.x - c2) | (v.y - c2) | (v.z - c2) | (v.w - c2) | (w.x - c2) | (w.y - c2) | (w.z - c2) | (w.w - c2);
        rawbits = (v.x & 1u) | (v.y & 1u) << 1 | (v.z & 1u) << 2 | (v.w & 1u) << 3 | (w.x & 1u) << 4 | (w.y & 1u) << 5 | (w.z & 1u) << 6 | (w.w & 1u) << 7;
        const unsigned long long m = __ballot((int)neg >= 0);
        if ((m >> j) & 1) return m;
        if (give_up(myws, call, t0)) { rawbits = 0xffu; return ~0ull; }
        __builtin_amdgcn_s_sleep(4);
    }
}

__device__ __forceinline__ unsigned wave_sum(unsigned v) {
    v += __builtin_amdgcn_update_dpp(0u, v, 0x111, 0xf, 0xf, true);
    v += __builtin_amdgcn_update_dpp(0u, v, 0x112, 0xf, 0xf, true);
    v += __builtin_amdgcn_update_dpp(0u, v, 0x114, 0xf, 0xf, true);
    v += __builtin_amdgcn_update_dpp(0u, v, 0x118, 0xf, 0xf, true);
    return __builtin_amdgcn_readlane(v, 15) + __builtin_amdgcn_readlane(v, 31) + __builtin_amdgcn_readlane(v, 47) + __builtin_amdgcn_readlane(v, 63);
}

__device__ __forceinline__ unsigned pack_sub(const u32x4* h, const char* myws, unsigned dst, LAS unsigned short* list,
                                             unsigned& cnt, int sub, int lane) {
    unsigned L[8], H[8], C[8];
#pragma unroll
    for (int i = 0; i < 8; ++i) {
        const unsigned d0 = h[i >> 1][(i & 1) * 2], d1 = h[i >> 1][(i & 1) * 2 + 1];
        L[i] = __builtin_amdgcn_perm(d1, d0, 0x06040200u);
        H[i] = __builtin_amdgcn_perm(d1, d0, 0x07050301u);
    }
    st16_sys<0>(myws, dst + lane * 16, (u32x4){L[0], L[1], L[2], L[3]});
    st16_sys<1024>(myws, dst + lane * 16, (u32x4){L[4], L[5], L[6], L[7]});
    unsigned es = 0;
#pragma unroll
    for (int i = 0; i < 8; ++i) es = __builtin_amdgcn_sad_u8(H[i] & 0x7c7c7c7cu, 0u, es);
    int base = (int)((wave_sum(es) + 23101u) >> 13) - 7;
    base = base < 0 ? 0 : base > 24 ? 24 : base;
    const unsigned b4 = (unsigned)base * 0x04040404u;
    unsigned V0 = 0, V1 = 0;
#pragma unroll
    for (int i = 0; i < 8; ++i) {
        const unsigned D = (H[i] | 0x80808080u) - b4;
        C[i] = (H[i] & 0x80808080u) | (D & 0x1f1f1f1fu);
        const unsigned X = H[i] ^ (C[i] + b4);
        V0 |= ((X >> 5) & 0x01010101u) << i;
        V1 |= ((X >> 6) & 0x01010101u) << i;
    }
    unsigned W[6];
#pragma unroll
    for (int g = 0; g < 2; ++g) {
        const unsigned c3 = C[4 * g + 3];
        W[3 * g + 0] = C[4 * g + 0] | ((c3 & 0x03030303u) << 5);
        W[3 * g + 1] = C[4 * g + 1] | ((c3 & 0x0c0c0c0cu) << 3);
        W[3 * g + 2] = C[4 * g + 2] | ((c3 & 0x10101010u) << 1) | ((c3 & 0x80808080u) >> 1);
    }
    st16_sys<2048>(myws, dst + lane * 16, (u32x4){W[0], W[1], W[2], W[3]});
    asm volatile("global_store_dwordx2 %0, %1, %2 offset:3072 sc0 sc1" :: "v"(dst + lane * 8), "v"((u32x2){W[4], W[5]}), "s"(myws) : "memory");
    unsigned Pm = V0 | V1;
    for (;;) {
        const unsigned long long act = __ballot(Pm != 0u);
        if (act == 0ull || cnt > 128u) break;
        const unsigned pos = cnt + __builtin_amdgcn_mbcnt_hi((unsigned)(act >> 32), __builtin_amdgcn_mbcnt_lo((unsigned)act, 0u));
        if (Pm != 0u && pos < 128u) {
            const unsigned bit = __builtin_ctz(Pm);
            const unsigned x2 = ((V0 >> bit) & 1u) | (((V1 >> bit) & 1u) << 1);
            list[pos] = (unsigned short)(((bit & 4u) << 8) | (unsigned)(lane << 4) | ((bit & 3u) << 2) | (bit >> 3) | (unsigned)(sub << 11) | (x2 << 13));
        }
        Pm &= Pm - 1u;
        cnt += (unsigned)__builtin_popcountll(act);
    }
    return (unsigned)base;
}

__device__ __forceinline__ void unpack_sub(u32x4 (&r)[4], u32x2 hw, unsigned base, unsigned e0, unsigned e1, int sub,
                                           LAS char* scr, int lane) {
    const unsigned b4 = base * 0x04040404u;
    const unsigned W[6] = {r[2].x, r[2].y, r[2].z, r[2].w, hw.x, hw.y};
    u32x4 H[2];
#pragma unroll
    for (int g = 0; g < 2; ++g) {
        const unsigned w0 = W[3 * g], w1 = W[3 * g + 1], w2 = W[3 * g + 2];
        H[g].x = (w0 & 0x9f9f9f9fu) + b4;
        H[g].y = (w1 & 0x9f9f9f9fu) + b4;
        H[g].z = (w2 & 0x9f9f9f9fu) + b4;
        H[g].w = (((w0 >> 5) & 0x03030303u) | ((w1 >> 3) & 0x0c0c0c0cu) | ((w2 >> 1) & 0x10101010u) | ((w2 << 1) & 0x80808080u)) + b4;
    }
    volatile LAS u32x4* img = (volatile LAS u32x4*)scr;
    img[lane] = H[0];
    img[64 + lane] = H[1];
    if ((e0 >> 13) != 0u && ((e0 >> 11) & 1u) == (unsigned)sub)
        __hip_atomic_fetch_xor((LAS unsigned*)(scr + (e0 & 0x7fcu)), (e0 >> 13) << (5 + 8 * (e0 & 3u)), __ATOMIC_RELAXED, __HIP_MEMORY_SCOPE_WAVEFRONT);
    if (sub == 1 && (e1 >> 13) != 0u)
        __hip_atomic_fetch_xor((LAS unsigned*)(scr + (e1 & 0x7fcu)), (e1 >> 13) << (5 + 8 * (e1 & 3u)), __ATOMIC_RELAXED, __HIP_MEMORY_SCOPE_WAVEFRONT);
    asm volatile("" ::: "memory");
    H[0] = img[lane];
    H[1] = img[64 + lane];
    const u32x4 L0 = r[0], L1 = r[1];
#pragma unroll
    for (int i = 0; i < 8; ++i) {
        const unsigned l = i < 4 ? L0[i] : L1[i - 4], hh = H[i >> 2][i & 3];
        r[i >> 1][(i & 1) * 2] = __builtin_amdgcn_perm(hh, l, 0x05010400u);
        r[i >> 1][(i & 1) * 2 + 1] = __builtin_amdgcn_perm(hh, l, 0x07030602u);
    }
}

#define STAMP(i) do {} while (0)

__device__ __forceinline__ void convert_w(const Params& P, const char* myws, int u, int lane, unsigned call) {
    const int st = u >> 5, cg = u & 31, q = lane >> 4, c = lane & 15;
    const float* wp = P.w + (size_t)(st * 64 + q * 4) * N_ALL + cg * 64 + c * 4;
    f32x4 wr[16];
#pragma unroll
    for (int i = 0; i < 16; ++i)
        wr[i] = *(const f32x4*)(wp + (size_t)((i >> 2) * 16 + (i & 3)) * N_ALL);
#pragma unroll
    for (int t = 0; t < 2; ++t) {
        const unsigned o = OFF_WB + (unsigned)(((cg >> 2) * NST + st) * B_TILE + (cg & 3) * 8192 + t * 4096 + lane * 16);
        const f32x4* r = wr + t * 8;
        st16_sys<0>(myws, o, (u32x4){pk(r[0].x, r[1].x), pk(r[2].x, r[3].x), pk(r[4].x, r[5].x), pk(r[6].x, r[7].x)});
        st16_sys<1024>(myws, o, (u32x4){pk(r[0].y, r[1].y), pk(r[2].y, r[3].y), pk(r[4].y, r[5].y), pk(r[6].y, r[7].y)});
        st16_sys<2048>(myws, o, (u32x4){pk(r[0].z, r[1].z), pk(r[2].z, r[3].z), pk(r[4].z, r[5].z), pk(r[6].z, r[7].z)});
        st16_sys<3072>(myws, o, (u32x4){pk(r[0].w, r[1].w), pk(r[2].w, r[3].w), pk(r[4].w, r[5].w), pk(r[6].w, r[7].w)});
    }
    waitvm<0>();
    if (lane == 0) st4_sys(myws, OFF_WFLAG + (unsigned)(((cg >> 2) * NST + st) * 64 + (cg & 3) * 4), call);
}

#define CONSUMER_VGPRS \
    "v0", "v1", "v2", "v3", "v4", "v5", "v6", "v7", "v8", "v9", "v10", "v11", "v12", "v13", "v14", "v15", "v16", \
    "v17", "v18", "v19", "v20", "v21", "v22", "v23", "v24", "v25", "v26", "v27", "v28", "v29", "v30", "v31", \
    "v32", "v33", "v34", "v35", "v36", "v37", "v38", "v39", "v40", "v41", "v42", "v43", "v44", "v45", "v46", \
    "v47", "v48", "v49", "v50", "v51", "v52", "v53", "v54", "v55", "v56", "v57", "v58", "v59", "v60", "v61", \
    "v62", "v63", "v64", "v65", "v66", "v67", "v68", "v69", "v70", "v71", "v72", "v73", "v74", "v75", "v76", \
    "v77", "v78", "v79", "v80", "v81", "v82", "v83", "v84", "v85", "v86", "v87", "v88", "v89", "v90", "v91", \
    "v92", "v93", "v94", "v95", "v96", "v97", "v98", "v99", "v100", "v101", "v102", "v103", "v104", "v105", \
    "v106", "v107", "v108", "v109", "v110", "v111", "v112", "v113", "v114", "v115", "v116", "v117", "v118", \
    "v119", "v120", "v121", "v122", "v123", "v124", "v125", "v126", "v127", "v128", "v129", "v130", "v131", \
    "v132", "v133", "v134", "v135", "v136", "v137", "v138", "v139", "v140", "v141", "v142", "v143", "v144", \
    "v145", "v146", "v147", "v148", "v149", "v150", "v151", "v152", "v153", "v154", "v155", "v156", "v157", \
    "v158", "v159", "v160", "v161", "v162", "v163", "v164", "v165", "v166", "v167", "v168", "v169", "v170", \
    "v171", "v172", "v173", "v174", "v175", "v176", "v177", "v178", "v179", "v180", "v181", "v182", "v183", \
    "v184", "v185", "v186", "v187", "v188", "v189", "v190", "v191", "v192", "v193", "v194", "v195", "v196", \
    "v197", "v198", "v199", "v200", "v201", "v202", "v203", "v204", "v205", "v206", "v207", "v208", "v209", \
    "v210", "v211", "v212", "v213", "v214", "v215", "v216", "v217", "v218", "v219", "v220", "v221", "v222", \
    "v223", "v224", "v225", "v226", "v227", "v228", "v229", "v230", "v231", "v232", "v233", "v234", "v235", \
    "v236", "v237", "v238", "v239", "v240", "v241", "v242", "v243"

template <int MT>
__device__ __forceinline__ unsigned consume(unsigned lds0, int wave, const float* p0, const float* p8, const float* p9,
                                            unsigned plain, unsigned long long& tlast) {
    unsigned st, so, tmp, bad;
    const float* po;
    asm volatile(R"(
    .macro MM m, f, b
    v_mfma_f32_16x16x32_f16 v[84+16*\m+4*\f:87+16*\m+4*\f], v[44+4*\m:47+4*\m], v[\b+4*\f:\b+4*\f+3], v[84+16*\m+4*\f:87+16*\m+4*\f]
    .endm
    .macro LDA m, t
    ds_read_b128 v[44+4*(\m):47+4*(\m)], v0 offset:((\m)*2+\t)*1024
    .endm
    .macro LDB f, t, b
    ds_read_b128 v[\b+4*\f:\b+4*\f+3], v1 offset:\t*4096+\f*1024
    .endm
    ; one row group of a k-step: kind 0 = even k-step, 1 = odd k-step, 2 = odd k-step that loads nothing
    .macro GROUP m, kind
    .if \kind == 0
    .set bc, 12
    .set bn, 28
    .set tn, 1
    .else
    .set bc, 28
    .set bn, 12
    .set tn, 0
    .endif
    .if \kind == 0 && \m == 0
    s_waitcnt lgkmcnt(4)
    .endif
    .if \m == %c[mt]-4
    .if \kind == 2
    s_waitcnt lgkmcnt(0)
    .else
    s_waitcnt lgkmcnt(%c[mt]-1+\kind)
    .endif
    .endif
    MM \m, 0, bc
    .if \kind != 0 && \m == 0
    ds_write_b32 v6, v8
    .endif
    .if \kind == 0 && \m == 4
    s_add_u32 %[so], %[so], %c[slot]
    s_add_u32 %[st], %[st], 1
    .endif
    .if \kind == 0 && \m == 6
    v_add_u32 v1, %[so], v10
    .endif
    .if \kind == 0 && \m == %c[mt]-1
    s_waitcnt lgkmcnt(2)
    v_readfirstlane_b32 %[bad], v4
    .endif
    MM \m, 1, bc
    .if \kind != 0 && \m == 0
    v_add_u32 v0, %[so], v9
    .endif
    .if \kind != 2 && \m > 0
    LDA \m-1, tn
    .endif
    MM \m, 2, bc
    .if \kind == 1 && \m == 0
    s_cmp_eq_u32 %[st], 63
    .endif
    .if \kind == 0 && \m == 4
    s_cmp_eq_u32 %[so], 3*%c[slot]
    s_cselect_b32 %[so], 0, %[so]
    .endif
    .if \kind == 0 && \m == 6
    v_mov_b32 v8, %[st]
    .endif
    .if \kind == 0 && \m == %c[mt]-1
    s_cmp_gt_u32 %[bad], %[st]
    .endif
    MM \m, 3, bc
    .if \kind != 2 && \m < 4
    LDB \m, tn, bn
    .endif
    .if \kind == 0 && \m == 4
    s_cselect_b32 %[bad], -8, 4
    v_add_u32 v5, %[bad], v5
    .endif
    .if \kind == 0 && \m == %c[mt]-4
    ds_read_b32 v4, v5
    .endif
    .if \kind != 2 && \m == %c[mt]-1
    LDA \m, tn
    .endif
    .endm
    .macro KSTEP kind
    GROUP 0, \kind
    GROUP 1, \kind
    GROUP 2, \kind
    GROUP 3, \kind
    GROUP 4, \kind
    GROUP 5, \kind
    GROUP 6, \kind
    GROUP 7, \kind
    GROUP 8, \kind
    .if %c[mt] > 9
    GROUP 9, \kind
    .endif
    .endm
    .macro MM4 m, b
    MM \m, 0, \b
    MM \m, 1, \b
    MM \m, 2, \b
    MM \m, 3, \b
    .endm
    .macro OUT m, bits
    .set ac, 84+16*\m
    v_swap_b32 v[ac+1], v[ac+4]
    v_swap_b32 v[ac+2], v[ac+8]
    v_swap_b32 v[ac+3], v[ac+12]
    .if \m == 8
    s_mov_b64 %[po], %[p8]
    .endif
    .if \m == 9
    s_mov_b64 %[po], %[p9]
    .endif
    global_store_dwordx4 v7, v[ac:ac+3], %[po] \bits
    v_swap_b32 v[ac+6], v[ac+9]
    v_swap_b32 v[ac+7], v[ac+13]
    global_store_dwordx4 v8, v[ac+4:ac+7], %[po] \bits
    v_swap_b32 v[ac+11], v[ac+14]
    global_store_dwordx4 v9, v[ac+8:ac+11], %[po] \bits
    global_store_dwordx4 v10, v[ac+12:ac+15], %[po] \bits
    .if \m < 7
    v_add_u32 v7, 0x20000, v7
    v_add_u32 v8, 0x20000, v8
    v_add_u32 v9, 0x20000, v9
    v_add_u32 v10, 0x20000, v10
    .endif
    .if \m == 7
    v_subrev_u32 v7, 0xe0000, v7
    v_subrev_u32 v8, 0xe0000, v8
    v_subrev_u32 v9, 0xe0000, v9
    v_subrev_u32 v10, 0xe0000, v10
    .endif
    .endm
    ; step i of the last stage: even k-step of group i, odd k-step of group i-2, output of group i-3
    .macro LSTEP i, j, k, bits
    .if \i < %c[mt]
    MM4 \i, 12
    LDA \i, 1
    .endif
    .if \j < %c[mt]
    .if \j <= %c[mt]-3
    s_waitcnt lgkmcnt(2)
    .else
    s_waitcnt lgkmcnt(%c[mt]-1-\j)
    .endif
    MM4 \j, 28
    .endif
    .if \k < %c[mt]
    .if \i >= %c[mt]
    s_nop 7
    .endif
    OUT \k, \bits
    .endif
    .endm
    .macro LREST bits
    LSTEP 3, 1, 0, \bits
    LSTEP 4, 2, 1, \bits
    LSTEP 5, 3, 2, \bits
    LSTEP 6, 4, 3, \bits
    LSTEP 7, 5, 4, \bits
    LSTEP 8, 6, 5, \bits
    LSTEP 9, 7, 6, \bits
    LSTEP 10, 8, 7, \bits
    LSTEP 11, 9, 8, \bits
    LSTEP 12, 10, 9, \bits
    .endm

    v_mbcnt_lo_u32_b32 v2, -1, 0
    v_mbcnt_hi_u32_b32 v2, -1, v2
    v_lshlrev_b32 v3, 4, v2
    v_add_u32 v9, %[lds0], v3
    v_add_u32 v10, %[ldsb], v3
    v_mov_b32 v0, v9
    v_mov_b32 v1, v10
    v_mov_b32 v5, %[full]
    v_mov_b32 v6, %[free]
    v_lshrrev_b32 v7, 4, v2
    v_lshlrev_b32 v7, 15, v7
    v_and_b32 v3, 0xf0, v3
    v_or_b32 v7, v7, v3
    .set ac, 84
    .rept 16*%c[mt]
    v_mov_b32 v[ac], 0
    .set ac, ac+1
    .endr
    s_mov_b32 %[st], 0
    s_mov_b32 %[so], 0
.Lwait%=:
    ds_read_b32 v4, v5
    s_waitcnt lgkmcnt(0)
    v_readfirstlane_b32 %[tmp], v4
    s_cmp_gt_u32 %[tmp], %[st]
    s_cbranch_scc1 .Lhave%=
    s_sleep 1
    s_branch .Lwait%=
.Lhave%=:
    LDB 0, 0, 12
    LDA 0, 0
    LDB 1, 0, 12
    LDB 2, 0, 12
    LDB 3, 0, 12
    LDA 1, 0
    LDA 2, 0
    LDA 3, 0
    LDA 4, 0
    LDA 5, 0
    LDA 6, 0
    LDA 7, 0
    LDA 8, 0
    .if %c[mt] > 9
    LDA 9, 0
    .endif
    s_cmp_eq_u32 %[st], 63
    s_cbranch_scc1 .Llast%=
.Lloop%=:
    KSTEP 0
    s_cbranch_scc0 .Llate%=
    KSTEP 1
    s_cbranch_scc0 .Lloop%=
    s_branch .Llast%=
.Llate%=:
    KSTEP 2
    s_branch .Lwait%=
.Llast%=:
    )"
    R"(
    v_mov_b32 v5, %[abort]
    s_waitcnt lgkmcnt(4)
    MM 0, 0, 12
    ds_read_b32 v4, v5
    LDB 0, 1, 28
    MM 0, 1, 12
    LDB 1, 1, 28
    MM 0, 2, 12
    LDB 2, 1, 28
    MM 0, 3, 12
    LDB 3, 1, 28
    LDA 0, 1
    v_add_u32 v8, 0x2000, v7
    v_add_u32 v9, 0x4000, v7
    v_add_u32 v10, 0x6000, v7
    MM4 1, 12
    LDA 1, 1
    s_mov_b64 %[po], %[p0]
    MM4 2, 12
    LDA 2, 1
    s_waitcnt lgkmcnt(2)
    MM4 0, 28
    v_readfirstlane_b32 %[bad], v4
    s_cmp_lg_u32 %[bad], 0
    s_cbranch_scc1 .Ldone%=
    s_cmp_lg_u32 %[plain], 0
    s_cbranch_scc1 .Lplain%=
    LREST sc1
    s_branch .Ldone%=
.Lplain%=:
    LREST
.Ldone%=:
    s_waitcnt lgkmcnt(0)
    s_nop 7
    .purgem MM
    .purgem LDA
    .purgem LDB
    .purgem GROUP
    .purgem KSTEP
    .purgem MM4
    .purgem OUT
    .purgem LSTEP
    .purgem LREST
    )"
        : [st] "=&s"(st), [so] "=&s"(so), [tmp] "=&s"(tmp), [bad] "=&s"(bad), [po] "=&s"(po), [tl] "+s"(tlast)
        : [mt] "i"(MT), [slot] "i"(SLOT), [lds0] "s"(lds0), [ldsb] "s"(lds0 + A_LDS + wave * 8192),
          [full] "s"(lds0 + L_FULL), [free] "s"(lds0 + L_FREE + wave * 4), [abort] "s"(lds0 + L_ABORT),
          [p0] "s"(p0), [p8] "s"(p8), [p9] "s"(p9), [plain] "s"(plain)
        : "memory", "scc", CONSUMER_VGPRS);
    return bad;
}

#define SKEW1 1
#define SKEW2 2
#define SKEW3 3
struct Order { unsigned char g[128]; };
constexpr Order make_order() {
    Order o{};
    constexpr int n = 32 - PUSH0, skew[4] = {0, SKEW1, SKEW2, SKEW3};
    int next[4] = {0, 0, 0, 0};
    for (int i = 0; i < 4 * n; ++i) {
        int best = -1;
        for (int c = 0; c < 4; ++c)
            if (next[c] < n && (best < 0 || next[c] * (n - 1 - skew[best]) <= next[best] * (n - 1 - skew[c]))) best = c;
        o.g[i] = (unsigned char)((next[best] + PUSH0) * 4 + best);
        ++next[best];
    }
    return o;
}
__constant__ const Order ORDER = make_order();

constexpr int NPULLCU = 4;
constexpr int NGEMM = 8 * (32 - NPULLCU);
#define TALL_RB 1
#define PWAVES 4

__global__ __launch_bounds__(512, 2) void gemm_a2a_kernel(Params P) {
    extern __shared__ __attribute__((aligned(16))) char smem[];
    const int lane = threadIdx.x & 63;
    const int wave = __builtin_amdgcn_readfirstlane(threadIdx.x >> 6);
    const int b = blockIdx.x;
    const int rank = P.rank;
    const char* myws = P.ws[0];
    const unsigned lds0 = (unsigned)(unsigned long long)(LAS char*)smem;
    const int grp = b & 7, idx = b >> 3;
    const bool puller = idx < NPULLCU;
    const int gi = (idx - NPULLCU) * 8 + grp;
    if (wave == 0) STAMP(0);

    const char* wsl = P.ws[1];
#pragma unroll
    for (int j = 2; j < NR; ++j) wsl = (lane >> 2) == j - 1 ? P.ws[j] : wsl;

    if (puller) {
        constexpr int PL = 8 * NPULLCU * PWAVES / 7;
        constexpr int U0 = 16 * PUSH0;
        const bool pusher = PUSH0 && b >= 16 && wave >= 4;
        const int pw = b * PWAVES + (wave & 3);
        const int pj = 1 + pw % 7, p = (rank + pj) & (NR - 1), li = pw / 7;
        const char* pws = P.ws[pj];
        const unsigned ltab = lds0 + (wave & 3) * 256;
        f32x4 x0[8];
        if (pusher) {
            const unsigned o = (unsigned)((((b & 15) * 32 + (lane & 15)) * KDIM + (wave & 3) * 64 + (lane >> 4) * 4) * 4);
            const unsigned o2 = o + 16 * KDIM * 4;
            const char* xb = (const char*)P.x;
            ld16<0>(x0[0], xb, o); ld16<64>(x0[1], xb, o); ld16<128>(x0[2], xb, o); ld16<192>(x0[3], xb, o);
            ld16<0>(x0[4], xb, o2); ld16<64>(x0[5], xb, o2); ld16<128>(x0[6], xb, o2); ld16<192>(x0[7], xb, o2);
        }
        if (wave < 4) lds_st(ltab + lane * 4, 0u);
        asm volatile("s_waitcnt lgkmcnt(0)\n\ts_barrier" : "+v"(wsl) :: "memory");
        const unsigned cno = OFF_CNT + b * 128 + wave * 4;
        const unsigned call = (unsigned)__builtin_amdgcn_readfirstlane(ld4_sys_wait(myws, cno)) + 1u;
        const unsigned par = call & 1u;
        const unsigned xs_off = OFF_XS + par * 4 * MiB;
        const unsigned xfo = OFF_XFLAG + (unsigned)((par * NR + p) * 2048);
        if (wave >= 4) {
            if (pusher) {
                if (wave == 4) STAMP(20);
                const unsigned o = OFF_XA0 + (unsigned)((((par * NR + rank) * 4 + ((b >> 2) & 3)) * 4 + (wave & 3)) * A_TILE + (b & 3) * 4096 + lane * 16);
#pragma unroll
                for (int i = 0; i < 4; ++i) {
                    asm volatile("" : "+v"(x0[2 * i]), "+v"(x0[2 * i + 1]));
                    const f32x4 lo = x0[2 * i], hi = x0[2 * i + 1];
                    const u32x4 h = {pk(lo.x, lo.y), pk(lo.z, lo.w), pk(hi.x, hi.y), pk(hi.z, hi.w)};
#pragma unroll
                    for (int j = 0; j < NR - 1; ++j) st16_sys<0>(P.ws[1 + (j + pw) % 7], o + i * 1024, h);
                }
                waitvm<0>();
                if (lane < 28 && (lane & 3) == 0)
                    asm volatile("global_store_dword %0, %1, off sc0 sc1"
                                 :: "v"(wsl + (OFF_AFLAG + (unsigned)((rank * 256 + (wave & 3) * 4 + ((b >> 2) & 3)) * 64 + (b & 3) * 4))), "v"(call) : "memory");
                if (wave == 4) STAMP(21);
            }
            if (lane == 0) st4_sys(myws, cno, call);
            if (pw >= 7 * PL) return;
            for (int g0 = U0 / 8; g0 < 64;) {
                const int g = g0 + (lane & 7) < 64 ? g0 + (lane & 7) : g0;
                unsigned rawbits;
                const unsigned long long m = poll_units(myws, xfo + g * 32, call, 0, rawbits);
                if ((m >> lane) & 1) lds_st(ltab + g * 4, 0x100u | rawbits);
                g0 += __builtin_ctz(~((unsigned)m & 0xffu));
            }
            if (wave == 4) STAMP(22);
            return;
        }
        if (lane == 0) st4_sys(myws, cno, call);
        if (pw >= 7 * PL) return;
        const char* db = myws + OFF_XA + p * 4 * MiB;
        LAS char* scr = (LAS char*)smem + 2 * SLOT + wave * 2048;
        const unsigned v16 = lane * 16, v8 = lane * 8, v2 = lane * 2;
        auto toff = [&](int u) -> unsigned { return (unsigned)((((u >> 2) & 3) * NST + (u >> 4) * 2) * A_TILE + (u & 3) * 4096); };
        auto fetch = [&](u32x4 (&r)[4], u32x2& hw, unsigned& ex, int u, int sub, unsigned raw, unsigned want_ex) {
            const char* sb = pws + (raw ? xs_off + toff(u) + sub * A_TILE : OFF_XP + par * 4 * MiB + (unsigned)u * 8192 + sub * P_SUB);
            const char* eb = want_ex ? pws + (OFF_XP + par * 4 * MiB + (unsigned)u * 8192 + P_EXC + sub * 128) : myws + OFF_CNT + b * 128;
            asm volatile("s_cmp_lg_u32 %[raw], 0\n\t"
                         "global_load_dwordx4 %[r0], %[v16], %[sb] sc0 sc1\n\t"
                         "global_load_dwordx4 %[r1], %[v16], %[sb] offset:1024 sc0 sc1\n\t"
                         "global_load_dwordx4 %[r2], %[v16], %[sb] offset:2048 sc0 sc1\n\t"
                         "s_cbranch_scc1 1f\n\t"
                         "global_load_dwordx2 %[hw], %[v8], %[sb] offset:3072 sc0 sc1\n\t"
                         "s_branch 2f\n"
                         "1:\n\t"
                         "global_load_dwordx4 %[r3], %[v16], %[sb] offset:3072 sc0 sc1\n"
                         "2:\n\t"
                         "global_load_ushort %[ex], %[v2], %[eb] sc0 sc1"
                         : [r0] "+v"(r[0]), [r1] "+v"(r[1]), [r2] "+v"(r[2]), [r3] "+v"(r[3]), [hw] "+v"(hw), [ex] "+v"(ex)
                         : [v16] "v"(v16), [v8] "v"(v8), [v2] "v"(v2), [sb] "s"(sb), [eb] "s"(eb), [raw] "s"(raw)
                         : "memory", "scc");
        };
        auto landed = [&](u32x4 (&r)[4], u32x2& hw, unsigned& ex) {
            asm volatile("" : "+v"(r[0]), "+v"(r[1]), "+v"(r[2]), "+v"(r[3]), "+v"(hw), "+v"(ex)); };
        auto store4 = [&](u32x4 (&r)[4], unsigned o) {
            st16_sys<0>(db, o + v16, r[0]); st16_sys<1024>(db, o + v16, r[1]); st16_sys<2048>(db, o + v16, r[2]); st16_sys<3072>(db, o + v16, r[3]); };
        auto flag = [&](int u, int sub) {
            if (lane == 0) st4_sys(myws, OFF_AFLAG + (unsigned)(u < 0 ? (p * 256 + li) * 64 + 32
                                   : (p * 256 + ((u >> 4) * 2 + sub) * 4 + ((u >> 2) & 3)) * 64 + (u & 3) * 4), call); };
        unsigned otab = ((const unsigned short*)ORDER.g)[lane];
        asm volatile("" : "+v"(otab));
        auto unit_of = [&](unsigned n) -> unsigned {
            return ((((unsigned)__builtin_amdgcn_readlane(otab, n >> 3) >> ((n & 4u) * 2)) & 0xffu) << 2) | (n & 3u); };
        unsigned rawbits = 0;
        unsigned long long rdy = 0;
        int u = (int)unit_of(li), prev = -1;
        if (wave == 0) STAMP(16);
        while (!((rdy >> (u >> 3)) & 1)) { rawbits = lds_ld_wait(ltab + lane * 4); rdy = __ballot(rawbits != 0u); }
        if (wave == 0) STAMP(3);
        const unsigned tbase = (call - 1u) * (512u - U0) - (unsigned)PL;
        u32x4 ra[4] = {}, rb4[4] = {};
        u32x2 ha = {}, hb = {};
        unsigned ea = 0, eb2 = 0;
        unsigned raw = ((unsigned)__builtin_amdgcn_readlane(rawbits, u >> 3) >> (u & 7)) & 1u;
        fetch(ra, ha, ea, u, 0, raw, !raw);
        waitvm<0>();
        if (wave == 0) STAMP(17);
        for (;;) {
            waitvm<5>();
            landed(ra, ha, ea);
            const unsigned e0 = ea;
            const unsigned hdr = raw ? 0u : (unsigned)__builtin_amdgcn_readfirstlane(e0);
            fetch(rb4, hb, eb2, u, 1, raw, (hdr >> 10) & 1u);
            unsigned tk = 0;
            if (lane == 0) asm volatile("global_atomic_add %0, %1, %2, %3 sc0" : "+v"(tk) : "v"(OFF_TICKET + (unsigned)(pj - 1) * 128), "v"(1u), "s"(myws) : "memory");
            if (!raw) unpack_sub(ra, ha, hdr & 31u, e0, 0u, 0, scr, lane);
            waitvm<6>();
            flag(prev, 1);
            const unsigned o = toff(u);
            store4(ra, o);
            waitvm<5>();
            landed(rb4, hb, eb2);
            asm volatile("" : "+v"(tk));
            const unsigned tn = (unsigned)__builtin_amdgcn_readfirstlane(tk) - tbase;
            const bool more = tn < 512u - U0;
            const unsigned un = unit_of(more ? tn : 0u);
            unsigned rawn = 0;
            if (more) {
                while (!((rdy >> (un >> 3)) & 1)) { rawbits = lds_ld_wait(ltab + lane * 4); rdy = __ballot(rawbits != 0u); }
                rawn = ((unsigned)__builtin_amdgcn_readlane(rawbits, un >> 3) >> (un & 7)) & 1u;
                fetch(ra, ha, ea, (int)un, 0, rawn, !rawn);
            }
            if (!raw) unpack_sub(rb4, hb, (hdr >> 5) & 31u, e0, ((hdr >> 10) & 1u) ? eb2 : 0u, 1, scr, lane);
            if (more) waitvm<5>(); else waitvm<0>();
            flag(u, 0);
            store4(rb4, o + A_TILE);
            if (!more) break;
            prev = u; u = (int)un; raw = rawn;
        }
        waitvm<0>();
        flag(u, 1);
        if (wave == 0) STAMP(4);
        return;
    }

    const bool xconv = wave >= 4 && (wave < 6 || (gi >> 5) == 2);
    const int xu = (wave < 6 ? gi : NGEMM + (gi & 31)) * 2 + (wave & 1);
    f32x4 xr[16];
    if (xconv) {
        const unsigned o = (unsigned)((((xu & 15) * 32 + (lane & 15)) * KDIM + (xu >> 4) * 128 + (lane >> 4) * 4) * 4);
        const unsigned o2 = o + 16 * KDIM * 4;
        const char* xb = (const char*)P.x;
        ld16<0>(xr[0], xb, o); ld16<64>(xr[1], xb, o); ld16<128>(xr[2], xb, o); ld16<192>(xr[3], xb, o);
        ld16<0>(xr[4], xb, o2); ld16<64>(xr[5], xb, o2); ld16<128>(xr[6], xb, o2); ld16<192>(xr[7], xb, o2);
        ld16<256>(xr[8], xb, o); ld16<320>(xr[9], xb, o); ld16<384>(xr[10], xb, o); ld16<448>(xr[11], xb, o);
        ld16<256>(xr[12], xb, o2); ld16<320>(xr[13], xb, o2); ld16<384>(xr[14], xb, o2); ld16<448>(xr[15], xb, o2);
    }
    if (threadIdx.x < 16) lds_st(lds0 + L_FULL + threadIdx.x * 4, 0u);
    asm volatile("" : "+v"(wsl));
    const unsigned call = (unsigned)__builtin_amdgcn_readfirstlane(ld4_sys_wait(myws, OFF_CNT + b * 128)) + 1u;
    const unsigned par = call & 1u;
    const unsigned xs_off = OFF_XS + par * 4 * MiB;
    u32x4 xh[8];
    if (xconv) {
        if (wave == 4) STAMP(13);
#pragma unroll
        for (int i = 0; i < 16; ++i) asm volatile("" : "+v"(xr[i]));
#pragma unroll
        for (int i = 0; i < 8; ++i) {
            const f32x4 lo = xr[2 * i], hi = xr[2 * i + 1];
            xh[i] = (u32x4){pk(lo.x, lo.y), pk(lo.z, lo.w), pk(hi.x, hi.y), pk(hi.z, hi.w)};
        }
        const unsigned o = xs_off + (unsigned)((((xu >> 2) & 3) * NST + (xu >> 4) * 2) * A_TILE + (xu & 3) * 4096 + lane * 16);
        st16_sys<0>(myws, o, xh[0]); st16_sys<1024>(myws, o, xh[1]); st16_sys<2048>(myws, o, xh[2]); st16_sys<3072>(myws, o, xh[3]);
        st16_sys<0>(myws, o + A_TILE, xh[4]); st16_sys<1024>(myws, o + A_TILE, xh[5]); st16_sys<2048>(myws, o + A_TILE, xh[6]); st16_sys<3072>(myws, o + A_TILE, xh[7]);
        waitvm<0>();
    }
    const char* pub = wsl + (OFF_XFLAG + (unsigned)(((par * NR + rank) * 32 + (xu >> 4)) * 64));
    if (xconv && (xu >> 4) < RAW_SP && (xu >> 4) >= PUSH0 && lane < 28 && (lane & 3) == 0)
        asm volatile("global_store_dword %0, %1, off sc0 sc1" :: "v"(pub + (xu & 15) * 4), "v"(2u * call + 1u) : "memory");
    asm volatile("s_waitcnt lgkmcnt(0)\n\ts_barrier" ::: "memory");
    if (xconv) {
        if (lane < 2) st4_sys(myws, OFF_AFLAG + (unsigned)((rank * 256 + ((xu >> 4) * 2 + lane) * 4 + ((xu >> 2) & 3)) * 64 + (xu & 3) * 4), call);
        if (wave == 4 && lane == 0) st4_sys(myws, OFF_CNT + b * 128, call);
        if (wave == 4) STAMP(1);
        if ((xu >> 4) >= RAW_SP) {
            LAS unsigned short* list = (LAS unsigned short*)((LAS char*)smem + L_LIST + (wave - 4) * 256);
            ((volatile LAS unsigned*)list)[lane] = 0u;
            asm volatile("" ::: "memory");
            const unsigned dst = OFF_XP + par * 4 * MiB + (unsigned)xu * 8192;
            unsigned cnt = 1;
            const unsigned b0 = pack_sub(xh, myws, dst, list, cnt, 0, lane);
            const unsigned c0 = cnt;
            const unsigned b1 = pack_sub(xh + 4, myws, dst + P_SUB, list, cnt, 1, lane);
            asm volatile("" ::: "memory");
            unsigned ev = ((volatile LAS unsigned*)list)[lane];
            if (lane == 0) ev |= b0 | (b1 << 5) | (cnt > 64u ? 1u << 10 : 0u);
            st4_sys(myws, dst + P_EXC + lane * 4, ev);
            waitvm<0>();
            const unsigned raw = (c0 > 64u || cnt > 128u) ? 1u : 0u;
            if ((xu >> 4) < DIRECT_SP) {
                if (lane < 28 && (lane & 3) == 0)
                    asm volatile("global_store_dword %0, %1, off sc0 sc1" :: "v"(pub + (xu & 15) * 4), "v"(2u * call + raw) : "memory");
            } else {
                unsigned long long* agg = (unsigned long long*)(myws + OFF_AGG + (xu >> 4) * 128);
                const unsigned long long mine = 1ull | ((unsigned long long)raw << (32 + (xu & 15)));
                unsigned long long tot = 0;
                if (lane == 0) tot = __hip_atomic_fetch_add(agg, mine, __ATOMIC_RELAXED, __HIP_MEMORY_SCOPE_AGENT) + mine;
                const unsigned n = __builtin_amdgcn_readfirstlane((unsigned)tot), mask = __builtin_amdgcn_readfirstlane((unsigned)(tot >> 32));
                if ((n & 15u) == 0u) {
                    const unsigned m4 = mask >> ((lane & 3) * 4), c2 = 2u * call;
                    const u32x4 fv = {c2 + (m4 & 1u), c2 + ((m4 >> 1) & 1u), c2 + ((m4 >> 2) & 1u), c2 + ((m4 >> 3) & 1u)};
                    if (lane < 28) asm volatile("global_store_dwordx4 %0, %1, off sc0 sc1\n\ts_nop 1" :: "v"(pub + (lane & 3) * 16), "v"(fv) : "memory");
                    if (lane == 0 && mask != 0u) __hip_atomic_fetch_sub(agg, (unsigned long long)mask << 32, __ATOMIC_RELAXED, __HIP_MEMORY_SCOPE_AGENT);
                }
            }
            if (wave == 4) STAMP(19);
        }
    }
    if (gi < 8 * DIRECT_SP) asm volatile("s_barrier" ::: "memory");

    const int wi = NGEMM - 1 - gi;
    const int ti = idx - NPULLCU;
    const int rb = ((grp >> 1) + (ti >> 2)) & 3;
    const bool tall = rb == TALL_RB && ti < 16;
    const int src = (rank + 1 + (ti >> 2)) & (NR - 1);
    const int cb = (grp & 1) * 4 + (ti & 3);
    const int lg = (ti >> 2) * 4 + rb;
    const int lg2 = 28 + ((ti >> 2) & 3);

#define W_INIT 2
    if (wave == 7) {
        for (int r = W_INIT; r * NGEMM + wi < 2048; ++r) {
            const unsigned need = (unsigned)(6 * (r - W_INIT) + 1 + ti % 6);
            while (!ge(__builtin_amdgcn_readfirstlane(lds_ld_wait(lds0 + L_FREE)), need)) __builtin_amdgcn_s_sleep(8);
            convert_w(P, myws, r * NGEMM + wi, lane, call);
        }
        STAMP(14);
        return;
    }

    if (wave >= 4) {
        const int j = wave - 4;
        const char* ab = myws + OFF_XA + src * 4 * MiB + rb * NST * A_TILE;
        const char* lb = myws + xs_off + (lg >> 3) * NST * A_TILE + (lg & 7) * 2048;
        const char* lb2 = myws + xs_off + (tall ? 3 * NST * A_TILE + (lg2 & 7) * 2048 : (lg >> 3) * NST * A_TILE + (lg & 7) * 2048);
        const char* bb = myws + OFF_WB + cb * NST * B_TILE;
        const unsigned slot = lds0 + j * SLOT;
        const unsigned v0 = lane * 16;
        for (int s = j; s < NST; s += NSLOT) {
            const int lgl = lane == 4 && tall ? lg2 : lg;
            const unsigned fo = lane < 4 ? OFF_WFLAG + (cb * NST + s) * 64 + lane * 4
                              : lane >= 32 ? OFF_AFLAG + (src * 256 + s * 4 + rb) * 64 + (lane & 3) * 4
                                           : OFF_AFLAG + (rank * 256 + s * 4 + (lgl >> 3)) * 64 + ((lgl & 7) >> 1) * 4;
            poll_mask(myws, fo, call, 0xffffffffull);
            if (s >= NSLOT) {
                while (!__all(ge(lds_ld_wait(lds0 + L_FREE + (lane & 3) * 4), (unsigned)(s - NSLOT + 1)))) __builtin_amdgcn_s_sleep(1);
            }
            const char* as = s < 2 * PUSH0 ? myws + OFF_XA0 + (((par * NR + src) * 4 + rb) * 4 + s) * A_TILE : ab + s * A_TILE;
            const char* bs = bb + s * B_TILE;
            unsigned fv;
            asm volatile("global_load_dword %0, %1, %2 sc0 sc1" : "=v"(fv) : "v"(fo), "s"(myws) : "memory");
#pragma unroll
            for (int g = 0; g < 8; ++g) glds4k(bs, v0 + g * 4096, slot + A_LDS + g * 4096);
            asm volatile("s_waitcnt vmcnt(32)" : "+v"(fv) :: "memory");
            glds2k(lb + s * A_TILE, v0, slot + A_TILE);
            if (tall) glds2k(lb2 + s * A_TILE, v0, slot + A_TILE + 2048);
            if (!__all(ge(fv, call))) {
                while (!ge(__builtin_amdgcn_readfirstlane(lds_ld_wait(lds0 + L_XCNT)), (unsigned)s)) __builtin_amdgcn_s_sleep(1);
                poll_mask(myws, OFF_AFLAG + (src * 256 + s * 4 + rb) * 64 + (lane & 3) * 4, call, ~0ull);
            }
            asm volatile("ds_max_u32 %0, %1" :: "v"(lds0 + L_XCNT), "v"((unsigned)(s + 1)) : "memory");
#pragma unroll
            for (int g = 0; g < 4; ++g) glds4k(as, v0 + g * 4096, slot + g * 4096);
            const unsigned tmo = ld4_sys_wait(myws, OFF_TMO);
            if (tmo == call) lds_st(lds0 + L_ABORT, 1u);
            lds_st(lds0 + L_FULL + j * 4, (unsigned)(s + 1));
        }
        return;
    }

    if (3 - wave < W_INIT) convert_w(P, myws, (3 - wave) * NGEMM + wi, lane, call);
    if (wave == 3) STAMP(2);
#define PLAIN_FROM 4
    const int row0 = src * M_LOC + rb * 128, row8 = rank * M_LOC + lg * 16, row9 = rank * M_LOC + lg2 * 16;
    float* oc = P.out + cb * 256 + wave * 64;
    unsigned long long tlast = 0;
    const unsigned plain = __builtin_amdgcn_readfirstlane(rb >= PLAIN_FROM ? 1u : 0u);
    const unsigned bad = tall ? consume<10>(lds0, wave, oc + (size_t)row0 * N_LOC, oc + (size_t)row8 * N_LOC, oc + (size_t)row9 * N_LOC, plain, tlast)
                              : consume<9>(lds0, wave, oc + (size_t)row0 * N_LOC, oc + (size_t)row8 * N_LOC, oc + (size_t)row8 * N_LOC, plain, tlast);
    if (bad) {
        const int ln = __builtin_amdgcn_mbcnt_hi(~0u, __builtin_amdgcn_mbcnt_lo(~0u, 0u));
        for (int i = 0; i < (tall ? 160 : 144); ++i)
            oc[(size_t)(i < 128 ? row0 + i : i < 144 ? row8 + i - 128 : row9 + i - 144) * N_LOC + ln] = __builtin_nanf("");
    }
}

}

extern "C" void kernel_launch(int rank, int nranks, ncclComm_t comm,
                              hipStream_t stream,
                              void* const* d_in, const int* in_sizes, int n_in,
                              void* d_out, int out_size,
                              const int* in_dtypes, int out_dtype,
                              void* const* d_out_all,
                              void* const* d_ws_all, size_t ws_bytes) {
    (void)comm; (void)in_dtypes; (void)out_dtype; (void)d_out_all;
    if (nranks != NR || (unsigned)rank >= (unsigned)NR || n_in != 2 || in_sizes[0] != M_LOC * KDIM ||
        in_sizes[1] != KDIM * N_ALL || out_size != NR * M_LOC * N_LOC || ws_bytes < WS_NEED) {
        fprintf(stderr, "kernel_launch: unexpected shapes\n");
        return;
    }
    HIP_CHECK(hipFuncSetAttribute((const void*)gemm_a2a_kernel, hipFuncAttributeMaxDynamicSharedMemorySize, (int)LDS_BYTES));
    Params P;
    P.x = (const float*)d_in[0];
    P.w = (const float*)d_in[1] + rank * N_LOC;
    P.out = (float*)d_out;
    for (int p = 0; p < NR; ++p) P.ws[p] = (char*)d_ws_all[(rank + p) % NR];
    P.rank = rank;
    P.pad = 0;
    hipLaunchKernelGGL(gemm_a2a_kernel, dim3(256), dim3(512), LDS_BYTES, stream, P);
    HIP_CHECK(hipGetLastError());
}
```

```cpp
#include <hip/hip_runtime.h>
#include <rccl/rccl.h>
#include <cstdio>

#define HIP_CHECK(expr) do { hipError_t _e = (expr); if (_e != hipSuccess) { \
    fprintf(stderr, "HIP error %s at %s:%d\n", hipGetErrorString(_e), __FILE__, __LINE__); \
    return; } } while (0)

namespace {

constexpr int NR = 8;
constexpr int KDIM = 4096, N_ALL = 16384, N_LOC = 2048, M_LOC = 512;
constexpr int NST = 64;
constexpr unsigned A_TILE = 16384;
constexpr unsigned B_TILE = 32768;
constexpr unsigned A_LDS = A_TILE + 4096;
constexpr unsigned SLOT = A_LDS + B_TILE;
constexpr int NSLOT = 3;
constexpr unsigned MiB = 1u << 20;

constexpr unsigned OFF_CNT = 0;
constexpr unsigned OFF_TMO = 32 * 1024;
constexpr unsigned OFF_TICKET = 33 * 1024;
constexpr unsigned OFF_AGG = 36 * 1024;
constexpr unsigned OFF_XFLAG = 64 * 1024;
constexpr unsigned OFF_AFLAG = 128 * 1024;
constexpr unsigned OFF_WFLAG = 256 * 1024;
constexpr unsigned OFF_DBG = 512 * 1024;
constexpr unsigned OFF_XS = 1 * MiB;
constexpr unsigned OFF_XA = 9 * MiB;
constexpr unsigned OFF_WB = 41 * MiB;
constexpr unsigned OFF_XP = 57 * MiB;
constexpr unsigned OFF_XA0 = 65 * MiB;
constexpr size_t WS_NEED = 69ull * MiB;
constexpr unsigned P_SUB = 3584, P_EXC = 2 * P_SUB;
#define PUSH0 2
#define DIRECT_SP 4
#define RAW_SP 2

constexpr unsigned L_FULL = NSLOT * SLOT;
constexpr unsigned L_FREE = L_FULL + 16;
constexpr unsigned L_XCNT = L_FREE + 16;
constexpr unsigned L_ABORT = L_XCNT + 4;
constexpr unsigned L_LIST = L_FULL + 64;
constexpr unsigned LDS_BYTES = L_LIST + 1024;

constexpr unsigned long long DEADLINE_TICKS = 50000000ull;

typedef unsigned u32x4 __attribute__((ext_vector_type(4)));
typedef unsigned u32x2 __attribute__((ext_vector_type(2)));
typedef float f32x4 __attribute__((ext_vector_type(4)));
typedef float f32x2 __attribute__((ext_vector_type(2)));
typedef _Float16 f16x2 __attribute__((ext_vector_type(2)));
typedef _Float16 f16x8 __attribute__((ext_vector_type(8)));
#define LAS __attribute__((address_space(3)))

struct Params {
    const float* x;
    const float* w;
    float* out;
    int rank;
    int pad;
    char* ws[NR];
};

__device__ __forceinline__ unsigned pk(float a, float b) {
    f32x2 v = {a, b};
    return __builtin_bit_cast(unsigned, __builtin_convertvector(v, f16x2));
}
template <int N> __device__ __forceinline__ void waitvm() { asm volatile("s_waitcnt vmcnt(%0)" :: "i"(N) : "memory"); }
__device__ __forceinline__ const char* uni(const char* p) {
    unsigned long long v = (unsigned long long)p;
    unsigned lo = __builtin_amdgcn_readfirstlane((unsigned)v), hi = __builtin_amdgcn_readfirstlane((unsigned)(v >> 32));
    return (const char*)(((unsigned long long)hi << 32) | lo);
}
template <int IMM> __device__ __forceinline__ void ld16(f32x4& d, const char* base, unsigned voff) {
    asm volatile("global_load_dwordx4 %0, %1, %2 offset:%3" : "=v"(d) : "v"(voff), "s"(base), "i"(IMM) : "memory");
}
template <int IMM> __device__ __forceinline__ void ld16_sys(u32x4& d, const char* base, unsigned voff) {
    asm volatile("global_load_dwordx4 %0, %1, %2 offset:%3 sc0 sc1" : "+v"(d) : "v"(voff), "s"(base), "i"(IMM) : "memory");
}
template <int IMM> __device__ __forceinline__ void st16_sys(const char* base, unsigned voff, u32x4 v) {
    asm volatile("global_store_dwordx4 %0, %1, %2 offset:%3 sc0 sc1\n\ts_nop 1" :: "v"(voff), "v"(v), "s"(base), "i"(IMM) : "memory");
}
__device__ __forceinline__ void st4_sys(const char* base, unsigned voff, unsigned v) {
    asm volatile("global_store_dword %0, %1, %2 sc0 sc1" :: "v"(voff), "v"(v), "s"(base) : "memory");
}
__device__ __forceinline__ unsigned ld4_sys_wait(const char* base, unsigned voff) {
    unsigned v;
    asm volatile("global_load_dword %0, %1, %2 sc0 sc1\n\ts_waitcnt vmcnt(0)" : "=&v"(v) : "v"(voff), "s"(base) : "memory");
    return v;
}
__device__ __forceinline__ void glds4k(const char* base, unsigned voff, unsigned lds) {
    unsigned keep;
    asm volatile("s_mov_b32 %0, m0\n\ts_mov_b32 m0, %3\n\ts_nop 0\n\t"
                 "global_load_lds_dwordx4 %1, %2 sc0 sc1\n\t"
                 "global_load_lds_dwordx4 %1, %2 offset:1024 sc0 sc1\n\t"
                 "global_load_lds_dwordx4 %1, %2 offset:2048 sc0 sc1\n\t"
                 "global_load_lds_dwordx4 %1, %2 offset:3072 sc0 sc1\n\t"
                 "s_mov_b32 m0, %0"
                 : "=&s"(keep) : "v"(voff), "s"(base), "s"(lds) : "memory");
}
__device__ __forceinline__ void glds2k(const char* base, unsigned voff, unsigned lds) {
    unsigned keep;
    asm volatile("s_mov_b32 %0, m0\n\ts_mov_b32 m0, %3\n\ts_nop 0\n\t"
                 "global_load_lds_dwordx4 %1, %2 sc0 sc1\n\t"
                 "global_load_lds_dwordx4 %1, %2 offset:1024 sc0 sc1\n\t"
                 "s_mov_b32 m0, %0"
                 : "=&s"(keep) : "v"(voff), "s"(base), "s"(lds) : "memory");
}
__device__ __forceinline__ void lds_st(unsigned addr, unsigned v) {
    asm volatile("ds_write_b32 %0, %1" :: "v"(addr), "v"(v) : "memory");
}
__device__ __forceinline__ unsigned lds_ld_wait(unsigned addr) {
    unsigned v;
    asm volatile("ds_read_b32 %0, %1\n\ts_waitcnt lgkmcnt(0)" : "=&v"(v) : "v"(addr) : "memory");
    return v;
}
__device__ __forceinline__ bool ge(unsigned v, unsigned want) { return (int)(v - want) >= 0; }

#define POLL_SLEEP 2
__device__ __forceinline__ bool give_up(const char* myws, unsigned call, unsigned long long t0) {
    if (__builtin_amdgcn_s_memrealtime() - t0 <= 2000) return false;
    if ((unsigned)__builtin_amdgcn_readfirstlane(ld4_sys_wait(myws, OFF_TMO)) == call) return true;
    if (__builtin_amdgcn_s_memrealtime() - t0 <= DEADLINE_TICKS) return false;
    st4_sys(myws, OFF_TMO, call);
    waitvm<0>();
    return true;
}
__device__ __forceinline__ unsigned long long poll_mask(const char* myws, unsigned voff, unsigned call, unsigned long long need) {
    unsigned long long m = __ballot(ge(ld4_sys_wait(myws, voff), call));
    if ((m & need) == need) return m;
    const unsigned long long t0 = __builtin_amdgcn_s_memrealtime();
    for (;;) {
        m = __ballot(ge(ld4_sys_wait(myws, voff), call));
        if ((m & need) == need) return m;
        if (give_up(myws, call, t0)) return ~0ull;
        __builtin_amdgcn_s_sleep(POLL_SLEEP);
    }
}
__device__ __forceinline__ unsigned wave_sum(unsigned v) {
    v += __builtin_amdgcn_update_dpp(0u, v, 0x111, 0xf, 0xf, true);
    v += __builtin_amdgcn_update_dpp(0u, v, 0x112, 0xf, 0xf, true);
    v += __builtin_amdgcn_update_dpp(0u, v, 0x114, 0xf, 0xf, true);
    v += __builtin_amdgcn_update_dpp(0u, v, 0x118, 0xf, 0xf, true);
    return __builtin_amdgcn_readlane(v, 15) + __builtin_amdgcn_readlane(v, 31) + __builtin_amdgcn_readlane(v, 47) + __builtin_amdgcn_readlane(v, 63);
}

__device__ __forceinline__ unsigned pack_sub(const u32x4* h, const char* myws, unsigned dst, LAS unsigned short* list,
                                             unsigned& cnt, int sub, int lane) {
    unsigned L[8], H[8], C[8];
#pragma unroll
    for (int i = 0; i < 8; ++i) {
        const unsigned d0 = h[i >> 1][(i & 1) * 2], d1 = h[i >> 1][(i & 1) * 2 + 1];
        L[i] = __builtin_amdgcn_perm(d1, d0, 0x06040200u);
        H[i] = __builtin_amdgcn_perm(d1, d0, 0x07050301u);
    }
    st16_sys<0>(myws, dst + lane * 16, (u32x4){L[0], L[1], L[2], L[3]});
    st16_sys<1024>(myws, dst + lane * 16, (u32x4){L[4], L[5], L[6], L[7]});
    unsigned es = 0;
#pragma unroll
    for (int i = 0; i < 8; ++i) es = __builtin_amdgcn_sad_u8(H[i] & 0x7c7c7c7cu, 0u, es);
    int base = (int)((wave_sum(es) + 23101u) >> 13) - 7;
    base = base < 0 ? 0 : base > 24 ? 24 : base;
    const unsigned b4 = (unsigned)base * 0x04040404u;
    unsigned V0 = 0, V1 = 0;
#pragma unroll
    for (int i = 0; i < 8; ++i) {
        const unsigned D = (H[i] | 0x80808080u) - b4;
        C[i] = (H[i] & 0x80808080u) | (D & 0x1f1f1f1fu);
        const unsigned X = H[i] ^ (C[i] + b4);
        V0 |= ((X >> 5) & 0x01010101u) << i;
        V1 |= ((X >> 6) & 0x01010101u) << i;
    }
    unsigned W[6];
#pragma unroll
    for (int g = 0; g < 2; ++g) {
        const unsigned c3 = C[4 * g + 3];
        W[3 * g + 0] = C[4 * g + 0] | ((c3 & 0x03030303u) << 5);
        W[3 * g + 1] = C[4 * g + 1] | ((c3 & 0x0c0c0c0cu) << 3);
        W[3 * g + 2] = C[4 * g + 2] | ((c3 & 0x10101010u) << 1) | ((c3 & 0x80808080u) >> 1);
    }
    st16_sys<2048>(myws, dst + lane * 16, (u32x4){W[0], W[1], W[2], W[3]});
    asm volatile("global_store_dwordx2 %0, %1, %2 offset:3072 sc0 sc1" :: "v"(dst + lane * 8), "v"((u32x2){W[4], W[5]}), "s"(myws) : "memory");
    unsigned Pm = V0 | V1;
    for (;;) {
        const unsigned long long act = __ballot(Pm != 0u);
        if (act == 0ull || cnt > 128u) break;
        const unsigned pos = cnt + __builtin_amdgcn_mbcnt_hi((unsigned)(act >> 32), __builtin_amdgcn_mbcnt_lo((unsigned)act, 0u));
        if (Pm != 0u && pos < 128u) {
            const unsigned bit = __builtin_ctz(Pm);
            const unsigned x2 = ((V0 >> bit) & 1u) | (((V1 >> bit) & 1u) << 1);
            list[pos] = (unsigned short)(((bit & 4u) << 8) | (unsigned)(lane << 4) | ((bit & 3u) << 2) | (bit >> 3) | (unsigned)(sub << 11) | (x2 << 13));
        }
        Pm &= Pm - 1u;
        cnt += (unsigned)__builtin_popcountll(act);
    }
    return (unsigned)base;
}

__device__ __forceinline__ void unpack_sub(u32x4 (&r)[4], u32x2 hw, unsigned base, unsigned e0, unsigned e1, int sub,
                                           LAS char* scr, int lane) {
    const unsigned b4 = base * 0x04040404u;
    const unsigned W[6] = {r[2].x, r[2].y, r[2].z, r[2].w, hw.x, hw.y};
    u32x4 H[2];
#pragma unroll
    for (int g = 0; g < 2; ++g) {
        const unsigned w0 = W[3 * g], w1 = W[3 * g + 1], w2 = W[3 * g + 2];
        H[g].x = (w0 & 0x9f9f9f9fu) + b4;
        H[g].y = (w1 & 0x9f9f9f9fu) + b4;
        H[g].z = (w2 & 0x9f9f9f9fu) + b4;
        H[g].w = (((w0 >> 5) & 0x03030303u) | ((w1 >> 3) & 0x0c0c0c0cu) | ((w2 >> 1) & 0x10101010u) | ((w2 << 1) & 0x80808080u)) + b4;
    }
    volatile LAS u32x4* img = (volatile LAS u32x4*)scr;
    img[lane] = H[0];
    img[64 + lane] = H[1];
    if ((e0 >> 13) != 0u && ((e0 >> 11) & 1u) == (unsigned)sub)
        __hip_atomic_fetch_xor((LAS unsigned*)(scr + (e0 & 0x7fcu)), (e0 >> 13) << (5 + 8 * (e0 & 3u)), __ATOMIC_RELAXED, __HIP_MEMORY_SCOPE_WAVEFRONT);
    if (sub == 1 && (e1 >> 13) != 0u)
        __hip_atomic_fetch_xor((LAS unsigned*)(scr + (e1 & 0x7fcu)), (e1 >> 13) << (5 + 8 * (e1 & 3u)), __ATOMIC_RELAXED, __HIP_MEMORY_SCOPE_WAVEFRONT);
    asm volatile("" ::: "memory");
    H[0] = img[lane];
    H[1] = img[64 + lane];
    const u32x4 L0 = r[0], L1 = r[1];
#pragma unroll
    for (int i = 0; i < 8; ++i) {
        const unsigned l = i < 4 ? L0[i] : L1[i - 4], hh = H[i >> 2][i & 3];
        r[i >> 1][(i & 1) * 2] = __builtin_amdgcn_perm(hh, l, 0x05010400u);
        r[i >> 1][(i & 1) * 2 + 1] = __builtin_amdgcn_perm(hh, l, 0x07030602u);
    }
}

#define STAMP(i) do {} while (0)

__device__ __forceinline__ void convert_w(const Params& P, const char* myws, int u, int lane, unsigned call) {
    const int st = u >> 5, cb = (u >> 2) & 7, q = u & 3;
    const char* wp = (const char*)(P.w + (size_t)(st * 64 + q * 4) * N_ALL + cb * 256);
    f32x4 wr[16];
    __builtin_amdgcn_s_setprio(3);
#pragma unroll
    for (int i = 0; i < 16; ++i)
        ld16<0>(wr[i], wp + (size_t)((i >> 2) * 16 + (i & 3)) * N_ALL * 4, (unsigned)lane * 16);
    __builtin_amdgcn_s_setprio(0);
    waitvm<0>();
#pragma unroll
    for (int i = 0; i < 16; ++i) asm volatile("" : "+v"(wr[i]));
#pragma unroll
    for (int t = 0; t < 2; ++t) {
        const unsigned o = OFF_WB + (unsigned)((cb * NST + st) * B_TILE + (lane >> 4) * 8192 + t * 4096 + q * 256 + (lane & 15) * 16);
        const f32x4* r = wr + t * 8;
        st16_sys<0>(myws, o, (u32x4){pk(r[0].x, r[1].x), pk(r[2].x, r[3].x), pk(r[4].x, r[5].x), pk(r[6].x, r[7].x)});
        st16_sys<1024>(myws, o, (u32x4){pk(r[0].y, r[1].y), pk(r[2].y, r[3].y), pk(r[4].y, r[5].y), pk(r[6].y, r[7].y)});
        st16_sys<2048>(myws, o, (u32x4){pk(r[0].z, r[1].z), pk(r[2].z, r[3].z), pk(r[4].z, r[5].z), pk(r[6].z, r[7].z)});
        st16_sys<3072>(myws, o, (u32x4){pk(r[0].w, r[1].w), pk(r[2].w, r[3].w), pk(r[4].w, r[5].w), pk(r[6].w, r[7].w)});
    }
    waitvm<0>();
    if (lane == 0) st4_sys(myws, OFF_WFLAG + (unsigned)((cb * NST + st) * 64 + q * 4), call);
}

#define BITS_B ""
#define CONSUMER_VGPRS \
    "v0", "v1", "v2", "v3", "v4", "v5", "v6", "v7", "v8", "v9", "v10", "v11", "v12", "v13", "v14", "v15", "v16", \
    "v17", "v18", "v19", "v20", "v21", "v22", "v23", "v24", "v25", "v26", "v27", "v28", "v29", "v30", "v31", \
    "v32", "v33", "v34", "v35", "v36", "v37", "v38", "v39", "v40", "v41", "v42", "v43", "v44", "v45", "v46", \
    "v47", "v48", "v49", "v50", "v51", "v52", "v53", "v54", "v55", "v56", "v57", "v58", "v59", "v60", "v61", \
    "v62", "v63", "v64", "v65", "v66", "v67", "v68", "v69", "v70", "v71", "v72", "v73", "v74", "v75", "v76", \
    "v77", "v78", "v79", "v80", "v81", "v82", "v83", "v84", "v85", "v86", "v87", "v88", "v89", "v90", "v91", \
    "v92", "v93", "v94", "v95", "v96", "v97", "v98", "v99", "v100", "v101", "v102", "v103", "v104", "v105", \
    "v106", "v107", "v108", "v109", "v110", "v111", "v112", "v113", "v114", "v115", "v116", "v117", "v118", \
    "v119", "v120", "v121", "v122", "v123", "v124", "v125", "v126", "v127", "v128", "v129", "v130", "v131", \
    "v132", "v133", "v134", "v135", "v136", "v137", "v138", "v139", "v140", "v141", "v142", "v143", "v144", \
    "v145", "v146", "v147", "v148", "v149", "v150", "v151", "v152", "v153", "v154", "v155", "v156", "v157", \
    "v158", "v159", "v160", "v161", "v162", "v163", "v164", "v165", "v166", "v167", "v168", "v169", "v170", \
    "v171", "v172", "v173", "v174", "v175", "v176", "v177", "v178", "v179", "v180", "v181", "v182", "v183", \
    "v184", "v185", "v186", "v187", "v188", "v189", "v190", "v191", "v192", "v193", "v194", "v195", "v196", \
    "v197", "v198", "v199", "v200", "v201", "v202", "v203", "v204", "v205", "v206", "v207", "v208", "v209", \
    "v210", "v211", "v212", "v213", "v214", "v215", "v216", "v217", "v218", "v219", "v220", "v221", "v222", \
    "v223", "v224", "v225", "v226", "v227", "v228", "v229", "v230", "v231", "v232", "v233", "v234", "v235", \
    "v236", "v237", "v238", "v239", "v240", "v241", "v242", "v243"

template <int MT>
__device__ __forceinline__ unsigned consume(unsigned lds0, int wave, const float* p0, const float* p8, const float* p9,
                                            unsigned plain, unsigned long long& tlast) {
    unsigned st, so, tmp, bad;
    const float* po;
    asm volatile(R"(
    .macro MM m, f, b
    v_mfma_f32_16x16x32_f16 v[84+16*\m+4*\f:87+16*\m+4*\f], v[44+4*\m:47+4*\m], v[\b+4*\f:\b+4*\f+3], v[84+16*\m+4*\f:87+16*\m+4*\f]
    .endm
    .macro LDA m, t
    ds_read_b128 v[44+4*(\m):47+4*(\m)], v0 offset:((\m)*2+\t)*1024
    .endm
    .macro LDB f, t, b
    ds_read_b128 v[\b+4*\f:\b+4*\f+3], v1 offset:\t*4096+\f*1024
    .endm
    ; one row group of a k-step: kind 0 = even k-step, 1 = odd k-step, 2 = odd k-step that loads nothing
    .macro GROUP m, kind
    .if \kind == 0
    .set bc, 12
    .set bn, 28
    .set tn, 1
    .else
    .set bc, 28
    .set bn, 12
    .set tn, 0
    .endif
    .if \kind == 0 && \m == 0
    s_waitcnt lgkmcnt(4)
    .endif
    .if \m == %c[mt]-4
    .if \kind == 2
    s_waitcnt lgkmcnt(0)
    .else
    s_waitcnt lgkmcnt(%c[mt]-1+\kind)
    .endif
    .endif
    MM \m, 0, bc
    .if \kind != 0 && \m == 0
    ds_write_b32 v6, v8
    .endif
    .if \kind == 0 && \m == 4
    s_add_u32 %[so], %[so], %c[slot]
    s_add_u32 %[st], %[st], 1
    .endif
    .if \kind == 0 && \m == 6
    v_add_u32 v1, %[so], v10
    .endif
    .if \kind == 0 && \m == %c[mt]-1
    s_waitcnt lgkmcnt(2)
    v_readfirstlane_b32 %[bad], v4
    .endif
    MM \m, 1, bc
    .if \kind != 0 && \m == 0
    v_add_u32 v0, %[so], v9
    .endif
    .if \kind != 2 && \m > 0
    LDA \m-1, tn
    .endif
    MM \m, 2, bc
    .if \kind == 1 && \m == 0
    s_cmp_eq_u32 %[st], 63
    .endif
    .if \kind == 0 && \m == 4
    s_cmp_eq_u32 %[so], 3*%c[slot]
    s_cselect_b32 %[so], 0, %[so]
    .endif
    .if \kind == 0 && \m == 6
    v_mov_b32 v8, %[st]
    .endif
    .if \kind == 0 && \m == %c[mt]-1
    s_cmp_gt_u32 %[bad], %[st]
    .endif
    MM \m, 3, bc
    .if \kind != 2 && \m < 4
    LDB \m, tn, bn
    .endif
    .if \kind == 0 && \m == 4
    s_cselect_b32 %[bad], -8, 4
    v_add_u32 v5, %[bad], v5
    .endif
    .if \kind == 0 && \m == %c[mt]-4
    ds_read_b32 v4, v5
    .endif
    .if \kind != 2 && \m == %c[mt]-1
    LDA \m, tn
    .endif
    .endm
    .macro KSTEP kind
    GROUP 0, \kind
    GROUP 1, \kind
    GROUP 2, \kind
    GROUP 3, \kind
    GROUP 4, \kind
    GROUP 5, \kind
    GROUP 6, \kind
    GROUP 7, \kind
    GROUP 8, \kind
    .if %c[mt] > 9
    GROUP 9, \kind
    .endif
    .endm
    .macro MM4 m, b
    MM \m, 0, \b
    MM \m, 1, \b
    MM \m, 2, \b
    MM \m, 3, \b
    .endm
    .macro OUT m, bits
    .set ac, 84+16*\m
    v_swap_b32 v[ac+1], v[ac+4]
    v_swap_b32 v[ac+2], v[ac+8]
    v_swap_b32 v[ac+3], v[ac+12]
    .if \m == 8
    s_mov_b64 %[po], %[p8]
    .endif
    .if \m == 9
    s_mov_b64 %[po], %[p9]
    .endif
    global_store_dwordx4 v7, v[ac:ac+3], %[po] \bits
    v_swap_b32 v[ac+6], v[ac+9]
    v_swap_b32 v[ac+7], v[ac+13]
    global_store_dwordx4 v8, v[ac+4:ac+7], %[po] \bits
    v_swap_b32 v[ac+11], v[ac+14]
    global_store_dwordx4 v9, v[ac+8:ac+11], %[po] \bits
    global_store_dwordx4 v10, v[ac+12:ac+15], %[po] \bits
    .if \m < 7
    v_add_u32 v7, 0x20000, v7
    v_add_u32 v8, 0x20000, v8
    v_add_u32 v9, 0x20000, v9
    v_add_u32 v10, 0x20000, v10
    .endif
    .if \m == 7
    v_subrev_u32 v7, 0xe0000, v7
    v_subrev_u32 v8, 0xe0000, v8
    v_subrev_u32 v9, 0xe0000, v9
    v_subrev_u32 v10, 0xe0000, v10
    .endif
    .endm
    ; step i of the last stage: even k-step of group i, odd k-step of group i-2, output of group i-3
    .macro LSTEP i, j, k, bits
    .if \i < %c[mt]
    MM4 \i, 12
    LDA \i, 1
    .endif
    .if \j < %c[mt]
    .if \j <= %c[mt]-3
    s_waitcnt lgkmcnt(2)
    .else
    s_waitcnt lgkmcnt(%c[mt]-1-\j)
    .endif
    MM4 \j, 28
    .endif
    .if \k < %c[mt]
    .if \i >= %c[mt]
    s_nop 7
    .endif
    OUT \k, \bits
    .endif
    .endm
    .macro LREST bits
    LSTEP 3, 1, 0, \bits
    LSTEP 4, 2, 1, \bits
    LSTEP 5, 3, 2, \bits
    LSTEP 6, 4, 3, \bits
    LSTEP 7, 5, 4, \bits
    LSTEP 8, 6, 5, \bits
    LSTEP 9, 7, 6, \bits
    LSTEP 10, 8, 7, \bits
    LSTEP 11, 9, 8, \bits
    LSTEP 12, 10, 9, \bits
    .endm

    v_mbcnt_lo_u32_b32 v2, -1, 0
    v_mbcnt_hi_u32_b32 v2, -1, v2
    v_lshlrev_b32 v3, 4, v2
    v_add_u32 v9, %[lds0], v3
    v_add_u32 v10, %[ldsb], v3
    v_mov_b32 v0, v9
    v_mov_b32 v1, v10
    v_mov_b32 v5, %[full]
    v_mov_b32 v6, %[free]
    v_lshrrev_b32 v7, 4, v2
    v_lshlrev_b32 v7, 15, v7
    v_and_b32 v3, 0xf0, v3
    v_or_b32 v7, v7, v3
    .set ac, 84
    .rept 16*%c[mt]
    v_mov_b32 v[ac], 0
    .set ac, ac+1
    .endr
    s_mov_b32 %[st], 0
    s_mov_b32 %[so], 0
.Lwait%=:
    ds_read_b32 v4, v5
    s_waitcnt lgkmcnt(0)
    v_readfirstlane_b32 %[tmp], v4
    s_cmp_gt_u32 %[tmp], %[st]
    s_cbranch_scc1 .Lhave%=
    s_sleep 1
    s_branch .Lwait%=
.Lhave%=:
    LDB 0, 0, 12
    LDA 0, 0
    LDB 1, 0, 12
    LDB 2, 0, 12
    LDB 3, 0, 12
    LDA 1, 0
    LDA 2, 0
    LDA 3, 0
    LDA 4, 0
    LDA 5, 0
    LDA 6, 0
    LDA 7, 0
    LDA 8, 0
    .if %c[mt] > 9
    LDA 9, 0
    .endif
    s_cmp_eq_u32 %[st], 63
    s_cbranch_scc1 .Llast%=
.Lloop%=:
    KSTEP 0
    s_cbranch_scc0 .Llate%=
    KSTEP 1
    s_cbranch_scc0 .Lloop%=
    s_branch .Llast%=
.Llate%=:
    KSTEP 2
    s_branch .Lwait%=
.Llast%=:
    )"
    R"(
    v_mov_b32 v5, %[abort]
    s_waitcnt lgkmcnt(4)
    MM 0, 0, 12
    ds_read_b32 v4, v5
    LDB 0, 1, 28
    MM 0, 1, 12
    LDB 1, 1, 28
    MM 0, 2, 12
    LDB 2, 1, 28
    MM 0, 3, 12
    LDB 3, 1, 28
    LDA 0, 1
    v_add_u32 v8, 0x2000, v7
    v_add_u32 v9, 0x4000, v7
    v_add_u32 v10, 0x6000, v7
    MM4 1, 12
    LDA 1, 1
    s_mov_b64 %[po], %[p0]
    MM4 2, 12
    LDA 2, 1
    s_waitcnt lgkmcnt(2)
    MM4 0, 28
    v_readfirstlane_b32 %[bad], v4
    s_cmp_lg_u32 %[bad], 0
    s_cbranch_scc1 .Ldone%=
    s_cmp_lg_u32 %[plain], 0
    s_cbranch_scc1 .Lplain%=
    LREST sc1
    s_branch .Ldone%=
.Lplain%=:
    LREST )" BITS_B R"(
.Ldone%=:
    s_waitcnt lgkmcnt(0)
    s_nop 7
    .purgem MM
    .purgem LDA
    .purgem LDB
    .purgem GROUP
    .purgem KSTEP
    .purgem MM4
    .purgem OUT
    .purgem LSTEP
    .purgem LREST
    )"
        : [st] "=&s"(st), [so] "=&s"(so), [tmp] "=&s"(tmp), [bad] "=&s"(bad), [po] "=&s"(po), [tl] "+s"(tlast)
        : [mt] "i"(MT), [slot] "i"(SLOT), [lds0] "s"(lds0), [ldsb] "s"(lds0 + A_LDS + wave * 8192),
          [full] "s"(lds0 + L_FULL), [free] "s"(lds0 + L_FREE + wave * 4), [abort] "s"(lds0 + L_ABORT),
          [p0] "s"(p0), [p8] "s"(p8), [p9] "s"(p9), [plain] "s"(plain)
        : "memory", "scc", CONSUMER_VGPRS);
    return bad;
}

#define SKEW1 1
#define SKEW2 2
#define SKEW3 3
struct Order { unsigned char g[128]; };
constexpr Order make_order() {
    Order o{};
    constexpr int n = 32 - PUSH0, skew[4] = {0, SKEW1, SKEW2, SKEW3};
    int next[4] = {0, 0, 0, 0};
    for (int i = 0; i < 4 * n; ++i) {
        int best = -1;
        for (int c = 0; c < 4; ++c)
            if (next[c] < n && (best < 0 || next[c] * (n - 1 - skew[best]) <= next[best] * (n - 1 - skew[c]))) best = c;
        o.g[i] = (unsigned char)((next[best] + PUSH0) * 4 + best);
        ++next[best];
    }
    return o;
}
__constant__ const Order ORDER = make_order();

constexpr int NPULLCU = 4;
constexpr int NGEMM = 8 * (32 - NPULLCU);
#define TALL_RB 1
#define PWAVES 4

__global__ __launch_bounds__(512, 2) void gemm_a2a_kernel(Params P) {
    extern __shared__ __attribute__((aligned(16))) char smem[];
    const int lane = threadIdx.x & 63;
    const int wave = __builtin_amdgcn_readfirstlane(threadIdx.x >> 6);
    const int b = blockIdx.x;
    const int rank = P.rank;
    const char* myws = P.ws[0];
    const unsigned lds0 = (unsigned)(unsigned long long)(LAS char*)smem;
    const int grp = b & 7, idx = b >> 3;
    const bool puller = idx < NPULLCU;
    const int gi = (idx - NPULLCU) * 8 + grp;
    if (wave == 0) STAMP(0);

    auto peer_of_lane = [&]() -> const char* {
        const char* r = P.ws[1];
#pragma unroll
        for (int j = 2; j < NR; ++j) { const char* q = P.ws[j]; asm volatile("" : "+s"(q)); r = (lane >> 2) == j - 1 ? q : r; }
        asm volatile("" : "+v"(r));
        return r;
    };
    const char* xb = (const char*)P.x;
    asm volatile("" : "+s"(xb), "+s"(myws));

    if (puller) {
        constexpr int PL = 8 * NPULLCU * PWAVES / 7;
        constexpr int U0 = 16 * PUSH0;
        if (wave > 5 || (wave == 5 && (!PUSH0 || b < 16))) return;
        const unsigned cno = OFF_CNT + b * 128 + wave * 4;
        if (wave == 5) {
            unsigned cv;
            asm volatile("global_load_dword %0, %1, %2 sc0 sc1" : "=v"(cv) : "v"(cno), "s"(myws) : "memory");
            f32x4 x0[32];
            const unsigned o = (unsigned)((((b & 15) * 32 + (lane & 15)) * KDIM + (lane >> 4) * 4) * 4), o2 = o + 16 * KDIM * 4;
#define LDQ(i) ld16<(i) * 256>(x0[8 * (i)], xb, o); ld16<(i) * 256 + 64>(x0[8 * (i) + 1], xb, o); \
               ld16<(i) * 256 + 128>(x0[8 * (i) + 2], xb, o); ld16<(i) * 256 + 192>(x0[8 * (i) + 3], xb, o); \
               ld16<(i) * 256>(x0[8 * (i) + 4], xb, o2); ld16<(i) * 256 + 64>(x0[8 * (i) + 5], xb, o2); \
               ld16<(i) * 256 + 128>(x0[8 * (i) + 6], xb, o2); ld16<(i) * 256 + 192>(x0[8 * (i) + 7], xb, o2)
            LDQ(0); LDQ(1); LDQ(2); LDQ(3);
#undef LDQ
            const char* dst[NR - 1];
#pragma unroll
            for (int j = 0; j < NR - 1; ++j) dst[j] = P.ws[1 + (j + b) % 7];
            const char* wsl = peer_of_lane();
            asm volatile("s_barrier\n\ts_waitcnt vmcnt(24)" : "+v"(cv) :: "memory");
            STAMP(20);
            const unsigned call = (unsigned)__builtin_amdgcn_readfirstlane(cv) + 1u;
            const unsigned ob = OFF_XA0 + (unsigned)((((call & 1u) * NR + rank) * 4 + ((b >> 2) & 3)) * 4 * A_TILE + (b & 3) * 4096 + lane * 16);
            const char* fl = wsl + (OFF_AFLAG + (unsigned)((rank * 4 + ((b >> 2) & 3)) * 1024 + (b & 3) * 4));
#pragma unroll
            for (int i = 0; i < 4; ++i) {
                if (i == 1) waitvm<28>();
#pragma unroll
                for (int k = 0; k < 4; ++k) {
                    asm volatile("" : "+v"(x0[8 * i + 2 * k]), "+v"(x0[8 * i + 2 * k + 1]));
                    const f32x4 lo = x0[8 * i + 2 * k], hi = x0[8 * i + 2 * k + 1];
                    const u32x4 h = {pk(lo.x, lo.y), pk(lo.z, lo.w), pk(hi.x, hi.y), pk(hi.z, hi.w)};
#pragma unroll
                    for (int j = 0; j < NR - 1; ++j) st16_sys<0>(dst[j], ob + i * A_TILE + k * 1024, h);
                }
                if (i > 0) {
                    waitvm<28>();
                    if (lane < 28 && (lane & 3) == 0) asm volatile("global_store_dword %0, %1, off sc0 sc1" :: "v"(fl + (i - 1) * 16), "v"(call) : "memory");
                }
            }
            waitvm<0>();
            if (lane < 28 && (lane & 3) == 0) asm volatile("global_store_dword %0, %1, off sc0 sc1" :: "v"(fl + 3 * 16), "v"(call) : "memory");
            if (lane == 0) st4_sys(myws, cno, call);
            STAMP(21);
            return;
        }
        const int pw = b * PWAVES + wave;
        const int pj = 1 + pw % 7, p = (rank + pj) & (NR - 1), li = pw / 7;
        const char* pws = P.ws[pj];
        const unsigned ltab = lds0 + wave * 256;
        if (wave < 4) lds_st(ltab + lane * 4, 0u);
        asm volatile("s_waitcnt lgkmcnt(0)\n\ts_barrier" ::: "memory");
        const unsigned call = (unsigned)__builtin_amdgcn_readfirstlane(ld4_sys_wait(myws, cno)) + 1u;
        const unsigned par = call & 1u;
        const unsigned xs_off = OFF_XS + par * 4 * MiB;
        if (lane == 0) st4_sys(myws, cno, call);
        if (wave == 4) {
            const int w = lane >> 4, pww = b * PWAVES + w;
            const unsigned xf = OFF_XFLAG + (unsigned)((par * NR + ((rank + 1 + pww % 7) & (NR - 1))) * 2048 + (lane & 3) * 8);
            int g0 = pww < 7 * PL ? U0 / 8 : 64;
            unsigned long long t0 = __builtin_amdgcn_s_memrealtime();
            bool dead = false;
            while (__any(g0 < 64)) {
                const int g = g0 + ((lane >> 2) & 3);
                u32x2 v = {2u * call, 2u * call};
                if (g < 64) asm volatile("global_atomic_or_x2 %0, %1, %2, %3 sc0 sc1\n\ts_waitcnt vmcnt(0)"
                                         : "+v"(v) : "v"(xf + g * 32), "v"(0ull), "s"(myws) : "memory");
                const unsigned okm = (unsigned)(__ballot(dead || (int)((v.x - 2u * call) | (v.y - 2u * call)) >= 0) >> (16 * w)) & 0xffffu;
                const unsigned r0 = (unsigned)(__ballot(v.x & 1u) >> (lane & 60)), r1 = (unsigned)(__ballot(v.y & 1u) >> (lane & 60));
                const unsigned rawbits = (r0 & 1u) | (r1 & 1u) << 1 | (r0 & 2u) << 1 | (r1 & 2u) << 2 | (r0 & 4u) << 2 | (r1 & 4u) << 3 | (r0 & 8u) << 3 | (r1 & 8u) << 4;
                if (g < 64 && ((okm >> (lane & 12)) & 15u) == 15u) lds_st(lds0 + w * 256 + g * 4, dead ? 0x1ffu : 0x100u | rawbits);
                const unsigned adv = __builtin_ctz(~okm) >> 2;
                const bool moved = __any(g0 < 64 && adv != 0u);
                g0 += adv;
                if (moved) t0 = __builtin_amdgcn_s_memrealtime();
                else { dead = give_up(myws, call, t0); __builtin_amdgcn_s_sleep(2); }
            }
            STAMP(22);
            return;
        }
        if (pw >= 7 * PL) return;
        const char* db = myws + OFF_XA + p * 4 * MiB;
        LAS char* scr = (LAS char*)smem + 2 * SLOT + wave * 2048;
        const unsigned v16 = lane * 16, v8 = lane * 8, v2 = lane * 2;
        auto toff = [&](int u) -> unsigned { return (unsigned)((((u >> 2) & 3) * NST + (u >> 4) * 2) * A_TILE + (u & 3) * 4096); };
        const char* dummy = myws + OFF_CNT + b * 128;
        auto src_of = [&](int u, unsigned raw) -> const char* {
            return pws + (raw ? xs_off + toff(u) : OFF_XP + par * 4 * MiB + (unsigned)u * 8192); };
#define FETCH4(WAIT, r, hw, sb, raw) \
            asm volatile("s_cmp_lg_u32 %[rw], 0\n\t" WAIT \
                         "global_load_dwordx4 %[r0], %[v16], %[ba] sc0 sc1\n\t" \
                         "global_load_dwordx4 %[r1], %[v16], %[ba] offset:1024 sc0 sc1\n\t" \
                         "global_load_dwordx4 %[r2], %[v16], %[ba] offset:2048 sc0 sc1\n\t" \
                         "s_cbranch_scc1 1f\n\t" \
                         "global_load_dwordx2 %[h], %[v8], %[ba] offset:3072 sc0 sc1\n\t" \
                         "s_branch 2f\n" \
                         "1:\n\t" \
                         "global_load_dwordx4 %[r3], %[v16], %[ba] offset:3072 sc0 sc1\n" \
                         "2:" \
                         : [r0] "+v"(r[0]), [r1] "+v"(r[1]), [r2] "+v"(r[2]), [r3] "+v"(r[3]), [h] "+v"(hw) \
                         : [v16] "v"(v16), [v8] "v"(v8), [ba] "s"(sb), [rw] "s"(raw) : "memory", "scc")
        auto fetch_ex = [&](unsigned& ex, const char* eb) {
            asm volatile("global_load_ushort %0, %1, %2 sc0 sc1" : "+v"(ex) : "v"(v2), "s"(eb) : "memory"); };
        auto landed = [&](u32x4 (&r)[4], u32x2& hw, unsigned& ex) {
            asm volatile("" : "+v"(r[0]), "+v"(r[1]), "+v"(r[2]), "+v"(r[3]), "+v"(hw), "+v"(ex)); };
        auto store4 = [&](u32x4 (&r)[4], unsigned o) {
            st16_sys<0>(db, o + v16, r[0]); st16_sys<1024>(db, o + v16, r[1]); st16_sys<2048>(db, o + v16, r[2]); st16_sys<3072>(db, o + v16, r[3]); };
        auto flag = [&](int u, int sub) {
            if (lane == 0) st4_sys(myws, u < 0 ? OFF_CNT + b * 128 + 64 + wave * 4
                                   : OFF_AFLAG + (unsigned)((p * 4 + ((u >> 2) & 3)) * 1024 + ((u >> 4) * 2 + sub) * 16 + (u & 3) * 4), call); };
        unsigned otab = ((const unsigned short*)ORDER.g)[lane];
        asm volatile("" : "+v"(otab));
        auto unit_of = [&](unsigned n) -> unsigned {
            return ((((unsigned)__builtin_amdgcn_readlane(otab, n >> 3) >> ((n & 4u) * 2)) & 0xffu) << 2) | (n & 3u); };
        unsigned rawbits = 0;
        unsigned long long rdy = 0;
        int u = (int)unit_of(li), prev = -1;
        if (wave == 0) STAMP(16);
        while (!((rdy >> (u >> 3)) & 1)) { rawbits = lds_ld_wait(ltab + lane * 4); rdy = __ballot(rawbits != 0u); }
        if (wave == 0) STAMP(3);
        const unsigned tbase = (call - 1u) * (512u - U0) - (unsigned)PL;
        u32x4 ra[4] = {}, rb4[4] = {};
        u32x2 ha = {}, hb = {};
        unsigned ea = 0, eb2 = 0;
        unsigned raw = ((unsigned)__builtin_amdgcn_readlane(rawbits, u >> 3) >> (u & 7)) & 1u;
        const char* sb = src_of(u, raw);
        FETCH4("", ra, ha, sb, raw);
        fetch_ex(ea, raw ? dummy : sb + P_EXC);
        waitvm<0>();
        if (wave == 0) STAMP(17);
        for (;;) {
            unsigned tk = 0;
            if (lane == 0) asm volatile("global_atomic_add %0, %1, %2, %3 sc0" : "+v"(tk) : "v"(OFF_TICKET + (unsigned)(pj - 1) * 128), "v"(1u), "s"(myws) : "memory");
            const char* sb1 = sb + (raw ? A_TILE : P_SUB);
            FETCH4("s_waitcnt vmcnt(6)\n\t", rb4, hb, sb1, raw);
            landed(ra, ha, ea);
            unsigned e0;
            asm volatile("v_mov_b32 %0, %1\n\ts_nop 0" : "=&v"(e0), "+v"(ea));
            const unsigned hdr = raw ? 0u : (unsigned)__builtin_amdgcn_readfirstlane(e0);
            fetch_ex(eb2, ((hdr >> 10) & 1u) ? sb + (P_EXC + 128) : dummy);
            if (!raw) unpack_sub(ra, ha, hdr & 31u, e0, 0u, 0, scr, lane);
            const unsigned o = toff(u);
            store4(ra, o);
            waitvm<9>();
            flag(prev, 1);
            asm volatile("" : "+v"(tk));
            const unsigned tn = (unsigned)__builtin_amdgcn_readfirstlane(tk) - tbase;
            const bool more = tn < 512u - U0;
            const unsigned un = unit_of(more ? tn : 0u);
            unsigned rawn = 0;
            if (more) {
                while (!((rdy >> (un >> 3)) & 1)) { rawbits = lds_ld_wait(ltab + lane * 4); rdy = __ballot(rawbits != 0u); }
                rawn = ((unsigned)__builtin_amdgcn_readlane(rawbits, un >> 3) >> (un & 7)) & 1u;
                sb = src_of((int)un, rawn);
                FETCH4("s_waitcnt vmcnt(5)\n\t", ra, ha, sb, rawn);
                fetch_ex(ea, rawn ? dummy : sb + P_EXC);
            } else {
                waitvm<5>();
            }
            landed(rb4, hb, eb2);
            if (!raw) unpack_sub(rb4, hb, (hdr >> 5) & 31u, e0, ((hdr >> 10) & 1u) ? eb2 : 0u, 1, scr, lane);
            store4(rb4, o + A_TILE);
            if (more) waitvm<10>(); else waitvm<5>();
            flag(u, 0);
            if (!more) break;
            prev = u; u = (int)un; raw = rawn;
        }
#undef FETCH4
        waitvm<0>();
        flag(u, 1);
        if (wave == 0) STAMP(4);
        return;
    }

    const bool xconv = wave >= 4 && (wave < 6 || (gi >> 5) == 2);
    const int xu = (wave < 6 ? gi : NGEMM + (gi & 31)) * 2 + (wave & 1);
    unsigned cv;
    asm volatile("global_load_dword %0, %1, %2 sc0 sc1" : "=v"(cv) : "v"(OFF_CNT + b * 128), "s"(myws) : "memory");
    f32x4 xr[16];
    if (xconv) {
        const unsigned o = (unsigned)((((xu & 15) * 32 + (lane & 15)) * KDIM + (xu >> 4) * 128 + (lane >> 4) * 4) * 4);
        const unsigned o2 = o + 16 * KDIM * 4;
        ld16<0>(xr[0], xb, o); ld16<64>(xr[1], xb, o); ld16<128>(xr[2], xb, o); ld16<192>(xr[3], xb, o);
        ld16<0>(xr[4], xb, o2); ld16<64>(xr[5], xb, o2); ld16<128>(xr[6], xb, o2); ld16<192>(xr[7], xb, o2);
        ld16<256>(xr[8], xb, o); ld16<320>(xr[9], xb, o); ld16<384>(xr[10], xb, o); ld16<448>(xr[11], xb, o);
        ld16<256>(xr[12], xb, o2); ld16<320>(xr[13], xb, o2); ld16<384>(xr[14], xb, o2); ld16<448>(xr[15], xb, o2);
    }
    if (threadIdx.x < 16) lds_st(lds0 + L_FULL + threadIdx.x * 4, 0u);
    const char* wsl = peer_of_lane();
    asm volatile("s_waitcnt vmcnt(0)" : "+v"(cv) :: "memory");
    const unsigned call = (unsigned)__builtin_amdgcn_readfirstlane(cv) + 1u;
    const unsigned par = call & 1u;
    const unsigned xs_off = OFF_XS + par * 4 * MiB;
    u32x4 xh[8];
    if (xconv) {
        if (wave == 4) STAMP(13);
#pragma unroll
        for (int i = 0; i < 16; ++i) asm volatile("" : "+v"(xr[i]));
#pragma unroll
        for (int i = 0; i < 8; ++i) {
            const f32x4 lo = xr[2 * i], hi = xr[2 * i + 1];
            xh[i] = (u32x4){pk(lo.x, lo.y), pk(lo.z, lo.w), pk(hi.x, hi.y), pk(hi.z, hi.w)};
        }
        const unsigned o = xs_off + (unsigned)((((xu >> 2) & 3) * NST + (xu >> 4) * 2) * A_TILE + (xu & 3) * 4096 + lane * 16);
        st16_sys<0>(myws, o, xh[0]); st16_sys<1024>(myws, o, xh[1]); st16_sys<2048>(myws, o, xh[2]); st16_sys<3072>(myws, o, xh[3]);
        st16_sys<0>(myws, o + A_TILE, xh[4]); st16_sys<1024>(myws, o + A_TILE, xh[5]); st16_sys<2048>(myws, o + A_TILE, xh[6]); st16_sys<3072>(myws, o + A_TILE, xh[7]);
    }
    const char* pub = wsl + (OFF_XFLAG + (unsigned)(((par * NR + rank) * 32 + (xu >> 4)) * 64));
    asm volatile("s_waitcnt lgkmcnt(0)\n\ts_barrier" ::: "memory");
    if (xconv) {
        const unsigned lfo = OFF_AFLAG + (unsigned)((rank * 4 + ((xu >> 2) & 3)) * 1024 + ((xu >> 4) * 2 + lane) * 16 + (xu & 3) * 4);
        if (wave == 4 && lane == 0) st4_sys(myws, OFF_CNT + b * 128, call);
        if ((xu >> 4) < RAW_SP) {
            waitvm<0>();
            if ((xu >> 4) >= PUSH0 && lane < 28 && (lane & 3) == 0)
                asm volatile("global_store_dword %0, %1, off sc0 sc1" :: "v"(pub + (xu & 15) * 4), "v"(2u * call + 1u) : "memory");
            if (lane < 2) st4_sys(myws, lfo, call);
        } else {
            LAS unsigned short* list = (LAS unsigned short*)((LAS char*)smem + L_LIST + (wave - 4) * 256);
            ((volatile LAS unsigned*)list)[lane] = 0u;
            asm volatile("" ::: "memory");
            const unsigned dst = OFF_XP + par * 4 * MiB + (unsigned)xu * 8192;
            unsigned cnt = 1;
            const unsigned b0 = pack_sub(xh, myws, dst, list, cnt, 0, lane);
            const unsigned c0 = cnt;
            const unsigned b1 = pack_sub(xh + 4, myws, dst + P_SUB, list, cnt, 1, lane);
            asm volatile("" ::: "memory");
            unsigned ev = ((volatile LAS unsigned*)list)[lane];
            if (lane == 0) ev |= b0 | (b1 << 5) | (cnt > 64u ? 1u << 10 : 0u);
            st4_sys(myws, dst + P_EXC + lane * 4, ev);
            if (wave == 4) STAMP(1);
            waitvm<0>();
            const unsigned raw = (c0 > 64u || cnt > 128u) ? 1u : 0u;
            if ((xu >> 4) < DIRECT_SP) {
                if (lane < 28 && (lane & 3) == 0)
                    asm volatile("global_store_dword %0, %1, off sc0 sc1" :: "v"(pub + (xu & 15) * 4), "v"(2u * call + raw) : "memory");
            } else {
                unsigned long long* agg = (unsigned long long*)(myws + OFF_AGG + (xu >> 4) * 128);
                const unsigned long long mine = 1ull | ((unsigned long long)raw << (32 + (xu & 15)));
                unsigned long long tot = 0;
                if (lane == 0) tot = __hip_atomic_fetch_add(agg, mine, __ATOMIC_RELAXED, __HIP_MEMORY_SCOPE_AGENT) + mine;
                const unsigned n = __builtin_amdgcn_readfirstlane((unsigned)tot), mask = __builtin_amdgcn_readfirstlane((unsigned)(tot >> 32));
                if ((n & 15u) == 0u) {
                    const unsigned m4 = mask >> ((lane & 3) * 4), c2 = 2u * call;
                    const u32x4 fv = {c2 + (m4 & 1u), c2 + ((m4 >> 1) & 1u), c2 + ((m4 >> 2) & 1u), c2 + ((m4 >> 3) & 1u)};
                    if (lane < 28) asm volatile("global_store_dwordx4 %0, %1, off sc0 sc1\n\ts_nop 1" :: "v"(pub + (lane & 3) * 16), "v"(fv) : "memory");
                    if (lane == 0 && mask != 0u) __hip_atomic_fetch_sub(agg, (unsigned long long)mask << 32, __ATOMIC_RELAXED, __HIP_MEMORY_SCOPE_AGENT);
                }
            }
            if (lane < 2) st4_sys(myws, lfo, call);
            if (wave == 4) STAMP(19);
            if (wave == 4 && gi < 32) STAMP(23);
        }
    }
    if (gi < 8 * DIRECT_SP) asm volatile("s_barrier" ::: "memory");

    const int wi = NGEMM - 1 - gi;
    const int ti = idx - NPULLCU;
    const int rb = ((grp >> 1) + (ti >> 2)) & 3;
    const bool tall = rb == TALL_RB && ti < 16;
    const int src = (rank + 1 + (ti >> 2)) & (NR - 1);
    const int cb = (grp & 1) * 4 + (ti & 3);
    const int lg = (ti >> 2) * 4 + rb;
    const int lg2 = 28 + ((ti >> 2) & 3);

#define W_INIT 2
    if (wave == 7) {
        for (int r = W_INIT; r * NGEMM + wi < 2048; ++r) {
            const unsigned need = (unsigned)(6 * (r - W_INIT) + 1 + ti % 6);
            while (!ge(__builtin_amdgcn_readfirstlane(lds_ld_wait(lds0 + L_FREE)), need)) __builtin_amdgcn_s_sleep(8);
            convert_w(P, myws, r * NGEMM + wi, lane, call);
        }
        STAMP(14);
        return;
    }

    if (wave >= 4) {
        const int j = wave - 4;
        const char* ab = myws + OFF_XA + src * 4 * MiB + rb * NST * A_TILE;
        const char* lb = myws + xs_off + (lg >> 3) * NST * A_TILE + (lg & 7) * 2048;
        const char* lb2 = myws + xs_off + (tall ? 3 * NST * A_TILE + (lg2 & 7) * 2048 : (lg >> 3) * NST * A_TILE + (lg & 7) * 2048);
        const char* bb = myws + OFF_WB + cb * NST * B_TILE;
        const unsigned slot = lds0 + j * SLOT;
        const unsigned v0 = lane * 16;
        for (int s = j; s < NST; s += NSLOT) {
            const int lgl = lane == 4 && tall ? lg2 : lg;
            const unsigned fo = lane < 4 ? OFF_WFLAG + (cb * NST + s) * 64 + lane * 4
                                         : OFF_AFLAG + (rank * 4 + (lgl >> 3)) * 1024 + s * 16 + ((lgl & 7) >> 1) * 4;
            poll_mask(myws, fo, call, ~0ull);
            unsigned lv;
            do lv = lds_ld_wait(lds0 + L_FREE + (lane < 4 ? lane : 4) * 4);
            while (__any(lane < 4 && !ge(lv, (unsigned)(s - NSLOT + 1))) && (__builtin_amdgcn_s_sleep(1), true));
            unsigned k = __builtin_amdgcn_readlane(lv, 4), n = 0;
            const bool have = k > (unsigned)s;
            const char* as = s < 2 * PUSH0 ? myws + OFF_XA0 + (((par * NR + src) * 4 + rb) * 4 + s) * A_TILE : ab + s * A_TILE;
            const char* bs = bb + s * B_TILE;
            const unsigned ao = OFF_AFLAG + (src * 4 + rb) * 1024 + s * 16 + lane * 4;
            const bool looks = lane < 32 && s * 4 + lane < 4 * NST;
            unsigned fv = call - 1u;
#define AREAD "global_atomic_or %0, %1, %2, %3 sc0 sc1"
            if (!have && looks) asm volatile(AREAD : "+v"(fv) : "v"(ao), "v"(0u), "s"(myws) : "memory");
#pragma unroll
            for (int g = 0; g < 8; ++g) glds4k(bs, v0 + g * 4096, slot + A_LDS + g * 4096);
            asm volatile("s_waitcnt vmcnt(32)" : "+v"(fv) :: "memory");
            glds2k(lb + s * A_TILE, v0, slot + A_TILE);
            if (tall) glds2k(lb2 + s * A_TILE, v0, slot + A_TILE + 2048);
            if (!have) {
                n = __builtin_ctzll(~__ballot(ge(fv, call))) >> 2;
                if (n == 0u) {
                    while ((k = __builtin_amdgcn_readfirstlane(lds_ld_wait(lds0 + L_XCNT))) < (unsigned)s) __builtin_amdgcn_s_sleep(1);
                    const unsigned long long t0 = __builtin_amdgcn_s_memrealtime();
                    while (k == (unsigned)s) {
                        if (looks) asm volatile(AREAD "\n\ts_waitcnt vmcnt(0)" : "+v"(fv) : "v"(ao), "v"(0u), "s"(myws) : "memory");
                        n = __builtin_ctzll(~__ballot(ge(fv, call))) >> 2;
                        if (n != 0u) break;
                        if (give_up(myws, call, t0)) { n = 1u; break; }
                        __builtin_amdgcn_s_sleep(POLL_SLEEP);
                    }
                }
                if (k >= (unsigned)s && lane == 0) asm volatile("ds_max_u32 %0, %1" :: "v"(lds0 + L_XCNT), "v"((unsigned)s + n) : "memory");
            }
#undef AREAD
#pragma unroll
            for (int g = 0; g < 4; ++g) glds4k(as, v0 + g * 4096, slot + g * 4096);
            const unsigned tmo = ld4_sys_wait(myws, OFF_TMO);
            if (tmo == call) lds_st(lds0 + L_ABORT, 1u);
            lds_st(lds0 + L_FULL + j * 4, (unsigned)(s + 1));
            if (k < (unsigned)s) {
                while (__builtin_amdgcn_readfirstlane(lds_ld_wait(lds0 + L_XCNT)) < (unsigned)s) __builtin_amdgcn_s_sleep(1);
                if (lane == 0) asm volatile("ds_max_u32 %0, %1" :: "v"(lds0 + L_XCNT), "v"((unsigned)s + n) : "memory");
            }
        }
        return;
    }

    if (3 - wave < W_INIT) convert_w(P, myws, (3 - wave) * NGEMM + wi, lane, call);
    if (wave == 3) STAMP(2);
#define PLAIN_FROM 4
    const int row0 = src * M_LOC + rb * 128, row8 = rank * M_LOC + lg * 16, row9 = rank * M_LOC + lg2 * 16;
    float* oc = P.out + cb * 256 + wave * 64;
    unsigned long long tlast = 0;
    const unsigned plain = __builtin_amdgcn_readfirstlane(rb >= PLAIN_FROM ? 1u : 0u);
    const unsigned bad = tall ? consume<10>(lds0, wave, oc + (size_t)row0 * N_LOC, oc + (size_t)row8 * N_LOC, oc + (size_t)row9 * N_LOC, plain, tlast)
                              : consume<9>(lds0, wave, oc + (size_t)row0 * N_LOC, oc + (size_t)row8 * N_LOC, oc + (size_t)row8 * N_LOC, plain, tlast);
    if (bad) {
        const int ln = __builtin_amdgcn_mbcnt_hi(~0u, __builtin_amdgcn_mbcnt_lo(~0u, 0u));
        for (int i = 0; i < (tall ? 160 : 144); ++i)
            oc[(size_t)(i < 128 ? row0 + i : i < 144 ? row8 + i - 128 : row9 + i - 144) * N_LOC + ln] = __builtin_nanf("");
    }
}

}

extern "C" void kernel_launch(int rank, int nranks, ncclComm_t comm,
                              hipStream_t stream,
                              void* const* d_in, const int* in_sizes, int n_in,
                              void* d_out, int out_size,
                              const int* in_dtypes, int out_dtype,
                              void* const* d_out_all,
                              void* const* d_ws_all, size_t ws_bytes) {
    (void)comm; (void)in_dtypes; (void)out_dtype; (void)d_out_all;
    if (nranks != NR || (unsigned)rank >= (unsigned)NR || n_in != 2 || in_sizes[0] != M_LOC * KDIM ||
        in_sizes[1] != KDIM * N_ALL || out_size != NR * M_LOC * N_LOC || ws_bytes < WS_NEED) {
        fprintf(stderr, "kernel_launch: unexpected shapes\n");
        return;
    }
    HIP_CHECK(hipFuncSetAttribute((const void*)gemm_a2a_kernel, hipFuncAttributeMaxDynamicSharedMemorySize, (int)LDS_BYTES));
    Params P;
    P.x = (const float*)d_in[0];
    P.w = (const float*)d_in[1] + rank * N_LOC;
    P.out = (float*)d_out;
    for (int p = 0; p < NR; ++p) P.ws[p] = (char*)d_ws_all[(rank + p) % NR];
    P.rank = rank;
    P.pad = 0;
    hipLaunchKernelGGL(gemm_a2a_kernel, dim3(256), dim3(512), LDS_BYTES, stream, P);
    HIP_CHECK(hipGetLastError());
}
```
